```python
import numpy as np

kernel_name = "gpu_gemm_ar_m4096_k4096_n4096_f32_none"


ABSMAX_RTOL = 0.001

COMPUTE_DTYPE = "float32"
EPILOGUE = "none"
IN_QUANT = None
OUT_QUANT = None
IN_DTYPES = {"x": "float32", "w_mat": "float32"}
WORLD_SIZE = 8

def setup_inputs():

    x = np.random.randn(4096, 4096).astype(np.float32)
    w_mat = (np.random.randn(4096, 4096) / np.sqrt(4096)).astype(np.float32)
    return {"x": x, "w_mat": w_mat}

def _epilogue(y):
    return y

def reference(x, w_mat):
    y = (x @ w_mat).astype(np.float32)
    return _epilogue(y)
```

```cpp
#include <hip/hip_runtime.h>
#include <rccl/rccl.h>
#include <cstdio>

#define HIP_CHECK(expr) do { hipError_t _e = (expr); if (_e != hipSuccess) { \
    fprintf(stderr, "HIP error %s at %s:%d\n", hipGetErrorString(_e), __FILE__, __LINE__); \
    return; } } while (0)

namespace {

typedef _Float16 h8 __attribute__((ext_vector_type(8)));
typedef _Float16 h4 __attribute__((ext_vector_type(4)));
typedef float f4 __attribute__((ext_vector_type(4)));
typedef unsigned u4 __attribute__((ext_vector_type(4)));
typedef unsigned u2 __attribute__((ext_vector_type(2)));
typedef __attribute__((address_space(1))) unsigned gu32;
typedef __attribute__((address_space(3))) char lchar;
typedef __amdgpu_buffer_rsrc_t rsrc_t;

constexpr int NR = 8;
constexpr int NCOMM = 16, NGEMM = 240, NWG = NCOMM + NGEMM;
constexpr unsigned SHARD_BYTES = 4096u * 512u * 4u;
constexpr unsigned OUT_BYTES = 4096u * 4096u * 4u;
constexpr unsigned COL_BYTES = 4096u * 16u;
constexpr unsigned OPER_BYTES = 512u * COL_BYTES;
constexpr unsigned BUF_BYTES = 2u * OPER_BYTES;
constexpr size_t CTRL_BYTES = 1u << 20;
constexpr size_t WS_NEED = CTRL_BYTES + 2ull * BUF_BYTES;
constexpr unsigned OFF_CALL = 0;
constexpr unsigned OFF_TMO = 0x8000;
constexpr unsigned OFF_OWN = 0x9000;
constexpr unsigned OFF_PROG = 0x10000;
constexpr unsigned long long DEADLINE = 30000000ull;

constexpr int TM = 288, TMS = 272, TN = 256;
constexpr unsigned A_COL = TM * 16, B_COL = TN * 16;
constexpr unsigned A_BYTES = 8 * A_COL, B_BYTES = 8 * B_COL, STAGE = A_BYTES + B_BYTES;
constexpr unsigned LDS_CTL = 2 * STAGE;
constexpr unsigned LDS_BYTES = LDS_CTL + 64;
constexpr unsigned CV_Q = 1024 + 32, CV_WAVE = 4 * CV_Q;

struct Params {
  const float* x;
  const float* w;
  float* out;
  char* ws[NR];
  int rank;
};

__device__ __forceinline__ rsrc_t mk_rsrc(const void* p, unsigned bytes) {
  return __builtin_amdgcn_make_buffer_rsrc((void*)p, (short)0, (int)bytes, 0x00020000);
}
__device__ __forceinline__ unsigned rfl(unsigned v) { return __builtin_amdgcn_readfirstlane(v); }

#define STAMP(i) do {} while (0)

struct Cv {
  rsrc_t src;
  unsigned voff, kbs;
  bool is_w;
};
__device__ __forceinline__ Cv cv_setup(const Params& p, unsigned unit128, unsigned lane) {
  Cv c;
  c.is_w = unit128 >= 64;
  const unsigned u = unit128 & 63;
  c.src = mk_rsrc(c.is_w ? p.w : p.x, SHARD_BYTES);
  c.voff = c.is_w ? (lane >> 4) * (8u * 16384u) + u * 256u + (lane & 15) * 16u
                  : (u * 64u + (lane >> 3)) * 2048u + (lane & 7) * 16u;
  c.kbs = c.is_w ? 32u * 16384u : 128u;
  return c;
}
__device__ __forceinline__ void cv_load(const Cv& c, unsigned kb, f4 (&v)[8]) {
#pragma unroll
  for (int i = 0; i < 8; ++i) v[i] = __builtin_bit_cast(f4, __builtin_amdgcn_raw_buffer_load_b128(c.src, c.voff, kb * c.kbs + i * 16384u, 0));
}
__device__ __forceinline__ void cv_lds(const Cv& c, const f4 (&v)[8], lchar* l, unsigned lane) {
  if (c.is_w) {
#pragma unroll
    for (int u = 0; u < 4; ++u) {
      h8 g;
#pragma unroll
      for (int i = 0; i < 8; ++i) g[i] = (_Float16)v[i][u];
      *(__attribute__((address_space(3))) h8*)(l + (lane >> 4) * CV_Q + (16 * u + (lane & 15)) * 16) = g;
    }
  } else {
#pragma unroll
    for (int i = 0; i < 8; ++i) {
      h4 g;
#pragma unroll
      for (int u = 0; u < 4; ++u) g[u] = (_Float16)v[i][u];
      *(__attribute__((address_space(3))) h4*)(l + ((lane & 7) >> 1) * CV_Q + (8 * i + (lane >> 3)) * 16 + (lane & 1) * 8) = g;
    }
  }
  __builtin_amdgcn_fence(__ATOMIC_SEQ_CST, "wavefront");
}
__device__ __forceinline__ u4 cv_read(lchar* l, int q, unsigned lane) {
  return *(__attribute__((address_space(3))) u4*)(l + q * CV_Q + lane * 16);
}

__device__ __forceinline__ unsigned wave_min(unsigned v) {
#pragma unroll
  for (int o = 1; o < 32; o <<= 1) v = min(v, (unsigned)__shfl_xor((int)v, o));
  return rfl(v);
}

__global__ void __launch_bounds__(512, 2) gemm_ar_kernel(Params p) {
  __shared__ __attribute__((aligned(16))) char smem[LDS_BYTES];
  lchar* lds = (lchar*)smem;
  const unsigned lane = threadIdx.x & 63;
  const unsigned wid = rfl(threadIdx.x >> 6);
  const unsigned me = (unsigned)p.rank & (NR - 1);
  char* myws = p.ws[me];
  gu32* callp = (gu32*)(myws + OFF_CALL + blockIdx.x * 128u);
  gu32* tmo = (gu32*)(myws + OFF_TMO);
  STAMP(0);

  if (blockIdx.x < NCOMM) {
    const unsigned g = blockIdx.x;
    const unsigned unit128 = (wid >> 2) * 64 + g * 4 + (wid & 3);
    const Cv cv = cv_setup(p, unit128, lane);
    f4 v[8];
    cv_load(cv, 0, v);
    const unsigned call = rfl(__hip_atomic_load(callp, __ATOMIC_RELAXED, __HIP_MEMORY_SCOPE_AGENT)) + 1;
    const unsigned par = call & 1;
    rsrc_t dst[7];
#pragma unroll
    for (int i = 0; i < 7; ++i) {
      unsigned jj = i + wid; jj = 1 + (jj >= 14 ? jj - 14 : jj >= 7 ? jj - 7 : jj);
      const unsigned d = (me + jj) & (NR - 1);
      const unsigned off = (cv.is_w ? OPER_BYTES : 0u) + (64u + 7u - jj) * COL_BYTES;
      dst[i] = mk_rsrc(p.ws[d] + CTRL_BYTES + (size_t)par * BUF_BYTES + off, (cv.is_w ? BUF_BYTES : OPER_BYTES) - off);
    }
    const unsigned vout = (unit128 & 63) * 1024u + lane * 16u;
    const unsigned fj = lane < 7 ? lane + 1 : 1;
    char* fbase = p.ws[0];
#pragma unroll
    for (int d = 1; d < NR; ++d) fbase = (((me + fj) & (NR - 1)) == (unsigned)d) ? p.ws[d] : fbase;
    gu32* flag = (gu32*)(fbase + OFF_PROG + ((7u - fj) * 16u + g) * 128u);
    const bool pub = (threadIdx.x < 7);
    lchar* l = lds + wid * CV_WAVE;
    for (unsigned kb = 0; kb < 16; ++kb) {
      cv_lds(cv, v, l, lane);
      if (kb < 15) cv_load(cv, kb + 1, v);
#pragma unroll
      for (int q = 0; q < 4; ++q) {
        const unsigned kg = kb * 4 + q;
        const u4 d = cv_read(l, q, lane);
#pragma unroll
        for (int i = 0; i < 7; ++i) __builtin_amdgcn_raw_buffer_store_b128(d, dst[i], vout + kg * (7u * COL_BYTES), 0, 17);
        asm volatile("s_waitcnt vmcnt(7)\n\ts_barrier" ::: "memory");
        if (pub) __hip_atomic_store(flag, call * 128u + kg, __ATOMIC_RELAXED, __HIP_MEMORY_SCOPE_SYSTEM);
      }
    }
    asm volatile("s_waitcnt vmcnt(0)\n\ts_barrier" ::: "memory");
    if (pub) __hip_atomic_store(flag, call * 128u + 64u, __ATOMIC_RELAXED, __HIP_MEMORY_SCOPE_SYSTEM);
    if (threadIdx.x == 0) __hip_atomic_store(callp, call, __ATOMIC_RELAXED, __HIP_MEMORY_SCOPE_AGENT);
    STAMP(1);
    return;
  }

  const unsigned gi = blockIdx.x - NCOMM;
  const unsigned xcd = gi & 7, lin = (xcd & 1) * 30 + (gi >> 3);
  const unsigned ti = lin >> 2, tc = (xcd >> 1) * 4 + (lin & 3);
  const unsigned row0 = ti * TMS, col0 = tc * TN;
  unsigned call;
  {
    const unsigned gw = gi * 8 + wid;
    lchar* l = lds + wid * CV_WAVE;
    f4 v[8];
    Cv cv = cv_setup(p, gw & 127, lane);
    cv_load(cv, gw >> 7, v);
    call = rfl(__hip_atomic_load(callp, __ATOMIC_RELAXED, __HIP_MEMORY_SCOPE_AGENT)) + 1;
    const rsrc_t own = mk_rsrc(myws + CTRL_BYTES + (size_t)(call & 1) * BUF_BYTES, BUF_BYTES);
    const unsigned vout = (cv.is_w ? OPER_BYTES : 0u) + (gw & 63) * 1024u + lane * 16u;
    cv_lds(cv, v, l, lane);
    if (gw < 128) cv_load(cv, 15, v);
#pragma unroll
    for (int q = 0; q < 4; ++q)
      __builtin_amdgcn_raw_buffer_store_b128(cv_read(l, q, lane), own, vout + ((gw >> 7) * 4 + q) * COL_BYTES, 0, 16);
    if (gw < 128) {
      cv_lds(cv, v, l, lane);
#pragma unroll
      for (int q = 0; q < 4; ++q)
        __builtin_amdgcn_raw_buffer_store_b128(cv_read(l, q, lane), own, vout + (60 + q) * COL_BYTES, 0, 16);
    }
    if (threadIdx.x == 0) *(__attribute__((address_space(3))) u2*)(lds + LDS_CTL) = u2{0u, 0u};
    asm volatile("s_waitcnt vmcnt(0) lgkmcnt(0)\n\ts_barrier" ::: "memory");
    if (threadIdx.x == 0) {
      __hip_atomic_fetch_add((gu32*)(myws + OFF_OWN + xcd * 128u), 1u, __ATOMIC_RELAXED, __HIP_MEMORY_SCOPE_AGENT);
      __hip_atomic_store(callp, call, __ATOMIC_RELAXED, __HIP_MEMORY_SCOPE_AGENT);
    }
  }
  STAMP(1);
  volatile __attribute__((address_space(3))) unsigned* ctl = (volatile __attribute__((address_space(3))) unsigned*)(lds + LDS_CTL);

  gu32* pollp;
  unsigned pj = 0;
  {
    const unsigned g0 = row0 >> 8;
    const unsigned wh = lane / 7;
    pj = lane - wh * 7;
    const unsigned g = wh == 0 ? g0 : wh == 1 ? g0 + 1 : tc;
    unsigned off = OFF_PROG + (pj * 16u + g) * 128u;
    if (lane >= 21) off = OFF_OWN + (lane & 7) * 128u;
    pollp = (gu32*)(myws + off);
  }
  auto tiles_avail = [&](unsigned v) -> unsigned {
    unsigned a;
    if (lane < 21) {
      int d = (int)(v - call * 128u);
      d = d < 0 ? 0 : d > 64 ? 64 : d;
      a = 64u + 7u * (unsigned)d + pj;
    } else {
      a = (int)(v - call * 30u) >= 0 ? 512u : 0u;
    }
    return wave_min(a) >> 3;
  };
  auto wait_tiles = [&](unsigned need) {
    if (wid == 0) {
      const unsigned long long t0 = __builtin_amdgcn_s_memrealtime();
      for (;;) {
        const unsigned a = tiles_avail(__hip_atomic_load(pollp, __ATOMIC_RELAXED, __HIP_MEMORY_SCOPE_SYSTEM));
        if (a >= need) { if (lane == 0) ctl[0] = a; break; }
        if (__builtin_amdgcn_s_memrealtime() - t0 > DEADLINE ||
            rfl(__hip_atomic_load(tmo, __ATOMIC_RELAXED, __HIP_MEMORY_SCOPE_AGENT)) == call) {
          if (lane == 0) {
            __hip_atomic_store(tmo, call, __ATOMIC_RELAXED, __HIP_MEMORY_SCOPE_AGENT);
            ctl[1] = 1; ctl[0] = 64;
          }
          break;
        }
        __builtin_amdgcn_s_sleep(1);
      }
    } else {
      while (rfl(ctl[0]) < need) __builtin_amdgcn_s_sleep(1);
    }
  };

  const rsrc_t stg = mk_rsrc(myws + CTRL_BYTES + (size_t)(call & 1) * BUF_BYTES, BUF_BYTES);
  unsigned dvo[9];
#pragma unroll
  for (int i = 0; i < 9; ++i) {
    const unsigned pc = wid + 8 * i;
    if (pc < 36) {
      const unsigned e = pc * 64 + lane, q = e / TM, r = e - q * TM;
      dvo[i] = q * COL_BYTES + (row0 + r) * 16u;
    } else {
      const unsigned e = (pc - 36) * 64 + lane, q = e >> 8, r = e & 255;
      dvo[i] = OPER_BYTES + q * COL_BYTES + (col0 + r) * 16u;
    }
  }
  const unsigned lds0 = (unsigned)(size_t)lds;
  auto dma = [&](unsigned t) {
    const unsigned m0 = lds0 + (t & 1) * STAGE + wid * 1024u;
    const unsigned so = t * (8u * COL_BYTES);
#pragma unroll
    for (int i = 0; i < 8; ++i)
      asm volatile("s_mov_b32 m0, %0\n\ts_nop 0\n\tbuffer_load_dwordx4 %1, %2, %3 offen sc1 lds"
                   :: "s"(m0 + i * 8192u), "v"(dvo[i]), "s"(stg), "s"(so) : "memory");
    if (wid < 4)
      asm volatile("s_mov_b32 m0, %0\n\ts_nop 0\n\tbuffer_load_dwordx4 %1, %2, %3 offen sc1 lds"
                   :: "s"(m0 + 8 * 8192u), "v"(dvo[8]), "s"(stg), "s"(so) : "memory");
  };

  const unsigned wr = wid >> 2, wc = wid & 3;
  const unsigned a_off = (lane >> 4) * A_COL + (wr * 144 + (lane & 15)) * 16;
  const unsigned b_off = A_BYTES + (lane >> 4) * B_COL + (wc * 64 + (lane & 15)) * 16;
  f4 acc[9][4];
#pragma unroll
  for (int m = 0; m < 9; ++m)
#pragma unroll
    for (int n = 0; n < 4; ++n) acc[m][n] = f4{0.f, 0.f, 0.f, 0.f};

  wait_tiles(1);
  STAMP(2);
  dma(0);
  for (unsigned t = 0; t < 64; ++t) {
    asm volatile("s_waitcnt vmcnt(0)\n\ts_barrier" ::: "memory");
    const unsigned need = t + 2;
    const bool more = t < 63;
    const bool early = more && rfl(ctl[0]) >= need;
    unsigned pv;
    if (wid == 0) pv = __hip_atomic_load(pollp, __ATOMIC_RELAXED, __HIP_MEMORY_SCOPE_SYSTEM);
    if (early) dma(t + 1);
    {
      lchar* sb = lds + (t & 1) * STAGE;
#pragma unroll
      for (int s = 0; s < 2; ++s) {
        h8 a[9], b[4];
#pragma unroll
        for (int n = 0; n < 4; ++n) b[n] = *(__attribute__((address_space(3))) h8*)(sb + b_off + s * 4 * B_COL + n * 256);
#pragma unroll
        for (int m = 0; m < 9; ++m) a[m] = *(__attribute__((address_space(3))) h8*)(sb + a_off + s * 4 * A_COL + m * 256);
#pragma unroll
        for (int m = 0; m < 9; ++m)
#pragma unroll
          for (int n = 0; n < 4; ++n) acc[m][n] = __builtin_amdgcn_mfma_f32_16x16x32_f16(a[m], b[n], acc[m][n], 0, 0, 0);
      }
    }
    if (wid == 0) {
      const unsigned a = tiles_avail(pv);
      if (lane == 0 && a > ctl[0]) ctl[0] = a;
    }
    if (more && !early) { wait_tiles(need); dma(t + 1); }
  }
  STAMP(3);
  const bool bad = rfl(ctl[1]) != 0;
  const rsrc_t ro = mk_rsrc(p.out, OUT_BYTES);
  const unsigned ovo = ((row0 + wr * 144 + 4 * (lane >> 4)) * 4096u + col0 + wc * 64 + 4 * (lane & 15)) * 4u;
  const int mcnt = (wr == 1 && ti < 14) ? 8 : 9;
#pragma unroll
  for (int m = 0; m < 9; ++m) {
    if (m < mcnt) {
#pragma unroll
      for (int j = 0; j < 4; ++j) {
        f4 o = f4{acc[m][0][j], acc[m][1][j], acc[m][2][j], acc[m][3][j]};
        if (bad) o = f4{__builtin_nanf(""), __builtin_nanf(""), __builtin_nanf(""), __builtin_nanf("")};
        __builtin_amdgcn_raw_buffer_store_b128(__builtin_bit_cast(u4, o), ro, ovo + (16 * m + j) * 16384u, 0, 0);
      }
    }
  }
}

}

extern "C" void kernel_launch(int rank, int nranks, ncclComm_t comm,
                              hipStream_t stream,
                              void* const* d_in, const int* in_sizes, int n_in,
                              void* d_out, int out_size,
                              const int* in_dtypes, int out_dtype,
                              void* const* d_out_all,
                              void* const* d_ws_all, size_t ws_bytes) {
  if (nranks != NR || (unsigned)rank >= (unsigned)NR || ws_bytes < WS_NEED || n_in != 2 ||
      in_sizes[0] != 4096 * 512 || in_sizes[1] != 4096 * 512 || out_size != 4096 * 4096) {
    fprintf(stderr, "kernel_launch: unexpected problem shape\n");
    return;
  }
  Params p;
  p.x = (const float*)d_in[0];
  p.w = (const float*)d_in[1];
  p.out = (float*)d_out;
  for (int i = 0; i < NR; ++i) p.ws[i] = (char*)d_ws_all[i];
  p.rank = rank;
  hipLaunchKernelGGL(gemm_ar_kernel, dim3(NWG), dim3(512), 0, stream, p);
  HIP_CHECK(hipGetLastError());
}
```

```cpp
#include <hip/hip_runtime.h>
#include <rccl/rccl.h>
#include <cstdio>

#define HIP_CHECK(expr) do { hipError_t _e = (expr); if (_e != hipSuccess) { \
    fprintf(stderr, "HIP error %s at %s:%d\n", hipGetErrorString(_e), __FILE__, __LINE__); \
    return; } } while (0)

#define OAUX 16
#define SLEEP 2
#define SKEW4 0
#define LOOK 2
#define PWG 32
#define PWV 2
#define NPUSH 4
#define EARLY 7
#define PACE 4
#define GDELAY0 30
#define GDELAY 30
#define PXTRA 0

namespace {

typedef _Float16 h8 __attribute__((ext_vector_type(8)));
typedef _Float16 h4 __attribute__((ext_vector_type(4)));
typedef float f4 __attribute__((ext_vector_type(4)));
typedef unsigned u4 __attribute__((ext_vector_type(4)));
typedef __attribute__((address_space(1))) unsigned gu32;
typedef __attribute__((address_space(3))) char lchar;
typedef __attribute__((address_space(3))) unsigned lu32;
typedef __amdgpu_buffer_rsrc_t rsrc_t;

constexpr int NR = 8;
constexpr int NCOMM = 32, NGEMM = 224, NWG = NCOMM + NGEMM;
constexpr unsigned SHARD_BYTES = 4096u * 512u * 4u;
constexpr unsigned OUT_BYTES = 4096u * 4096u * 4u;
constexpr unsigned COL_BYTES = 4096u * 16u;
constexpr unsigned OPER_BYTES = 512u * COL_BYTES;
constexpr unsigned BUF_BYTES = 2u * OPER_BYTES;
constexpr size_t CTRL_BYTES = 1u << 20;
constexpr size_t WS_NEED = CTRL_BYTES + 2ull * BUF_BYTES;
constexpr unsigned COL_A = 64, COL_B = 64 + 7 * NPUSH;
constexpr unsigned OFF_CALL = 0;
constexpr unsigned OFF_TMO = 0x8000;
constexpr unsigned OFF_QUEUE = 0x8100;
constexpr unsigned OFF_RDY = 0x8200;
constexpr unsigned OFF_RDY0 = 0x20000;
constexpr unsigned OFF_TOP = 0x8500;
constexpr unsigned OFF_GO = 0x8700;
constexpr unsigned OFF_PSH = 0x8400;
constexpr unsigned OFF_CNT1 = 0x8600;
constexpr unsigned OFF_OWN = 0x9000;
constexpr unsigned OFF_DUMMY = 0xC000;
constexpr unsigned OFF_FB = 0x10000;
constexpr unsigned FB_U = 384;
constexpr unsigned long long DEADLINE = 30000000ull;

constexpr int TM = 304, TN = 256;
constexpr unsigned A_COL = TM * 16, B_COL = TN * 16;
constexpr unsigned A_BYTES = 4 * A_COL, B_BYTES = 4 * B_COL, STAGE = A_BYTES + B_BYTES;
constexpr unsigned LDS_CTL = 4 * STAGE;
constexpr unsigned LDS_BYTES = LDS_CTL + 64;
constexpr unsigned CV_Q = 1024 + 32, CV_WAVE = 4 * CV_Q;
constexpr unsigned LDS_TAB = 8 * CV_WAVE;

constexpr int NKG = 64 - NPUSH, NBLK = NKG * 8, NT = NBLK * 16;
constexpr int NP = PWG * PWV + PXTRA;
struct Order { unsigned short t[512]; };
constexpr Order make_order() {
  Order o{};
  int n = 0;
  for (int key = 0; key < NKG * 4 + 3 * SKEW4 + 4; ++key)
    for (int b = 0; b < 8; ++b) {
      const int r = key - (b >= 4 ? (b - 4) * SKEW4 : 0);
      if (r >= 0 && r < NKG * 4 && r % 4 == 0) o.t[n++] = (unsigned short)((r / 4) * 8 + b);
    }
  return o;
}
__constant__ const Order c_order = make_order();

struct Params {
  const float* x;
  const float* w;
  float* out;
  char* me;
  char* peer[7];
};

__device__ __forceinline__ rsrc_t mk_rsrc(const void* p, unsigned bytes) {
  return __builtin_amdgcn_make_buffer_rsrc((void*)p, (short)0, (int)bytes, 0x00020000);
}
__device__ __forceinline__ unsigned rfl(unsigned v) { return __builtin_amdgcn_readfirstlane(v); }

#define STAMPW(i, w) do {} while (0)
#define STAMPC(i) do {} while (0)
#define STAMP(i) STAMPW(i, 7)
#define TICK(x)
#define PUT(i, x)

template <class F>
__device__ __forceinline__ bool spin_until(char* me, unsigned call, unsigned lane, F cond) {
  gu32* tmo = (gu32*)(me + OFF_TMO);
  const unsigned long long t0 = __builtin_amdgcn_s_memrealtime();
  for (unsigned spin = 1;; ++spin) {
    if (cond()) return true;
    if ((spin & 63) == 0 && (__builtin_amdgcn_s_memrealtime() - t0 > DEADLINE ||
        rfl(__hip_atomic_load(tmo, __ATOMIC_RELAXED, __HIP_MEMORY_SCOPE_AGENT)) == call)) {
      if (lane == 0) __hip_atomic_store(tmo, call, __ATOMIC_RELAXED, __HIP_MEMORY_SCOPE_AGENT);
      return false;
    }
    __builtin_amdgcn_s_sleep(SLEEP);
  }
}

__global__ void __launch_bounds__(512, 2) gemm_ar_kernel(Params p) {
  __shared__ __attribute__((aligned(16))) char smem[LDS_BYTES];
  TICK(tk0);
  lchar* lds = (lchar*)smem;
  char* const me = p.me;
  float* const outp = p.out;
  const float* const xp = p.x;
  const float* const wp = p.w;
  const unsigned lane = threadIdx.x & 63;
  const unsigned wid = rfl(threadIdx.x >> 6);
  const unsigned prot = (blockIdx.x * 4 + wid) % 7;
  char* pptr[7];
  gu32* callp = (gu32*)(me + OFF_CALL + blockIdx.x * 128u);
  volatile lu32* ctl = (volatile lu32*)(lds + LDS_CTL);
  STAMP(0);
  const unsigned ctask = blockIdx.x < NCOMM ? (wid < 4 ? 128u : 0u) + blockIdx.x * 4 + (wid & 3)
                                            : 256u + wid * NGEMM + (blockIdx.x - NCOMM);
  const unsigned kb = ctask >> 7;
  const unsigned unit = ctask & 63;
  const bool is_w = (ctask & 64) != 0;
  lchar* cvl = lds + wid * CV_WAVE;
  unsigned call;
  unsigned long long tk1, tk2, tk3;
  {
    const rsrc_t src = mk_rsrc(is_w ? wp : xp, SHARD_BYTES);
    const unsigned voff = is_w ? (kb * 32u + (lane >> 4) * 8u) * 16384u + unit * 256u + (lane & 15) * 16u
                               : (unit * 64u + (lane >> 3)) * 2048u + kb * 128u + (lane & 7) * 16u;
    f4 v[8];
    if (kb != 1) {
      if (threadIdx.x == 448) { ctl[1] = 0; ctl[2] = 0; ctl[3] = 0; }
      asm volatile("s_waitcnt lgkmcnt(0)\n\ts_barrier" ::: "memory");
      __builtin_amdgcn_s_sleep(GDELAY0);
      for (unsigned k = 3; k < kb; k += 2) __builtin_amdgcn_s_sleep(GDELAY);
    }
#pragma unroll
    for (int i = 0; i < 8; ++i)
      v[i] = __builtin_bit_cast(f4, __builtin_amdgcn_raw_buffer_load_b128(src, voff, i * 16384u, 0));
    const unsigned callv = __hip_atomic_load(callp, __ATOMIC_RELAXED, __HIP_MEMORY_SCOPE_AGENT);
    if (kb == 1) asm volatile("s_barrier" ::: "memory");
#pragma unroll
    for (int i = 0; i < 7; ++i) pptr[i] = p.peer[prot + i >= 7 ? prot + i - 7 : prot + i];
    if (is_w) {
#pragma unroll
      for (int u = 0; u < 4; ++u) {
        h8 g;
#pragma unroll
        for (int i = 0; i < 8; ++i) g[i] = (_Float16)v[i][u];
        *(__attribute__((address_space(3))) h8*)(cvl + (lane >> 4) * CV_Q + (16 * u + (lane & 15)) * 16) = g;
      }
    } else {
#pragma unroll
      for (int i = 0; i < 8; ++i) {
        h4 g;
#pragma unroll
        for (int u = 0; u < 4; ++u) g[u] = (_Float16)v[i][u];
        *(__attribute__((address_space(3))) h4*)(cvl + ((lane & 7) >> 1) * CV_Q + (8 * i + (lane >> 3)) * 16 + (lane & 1) * 8) = g;
      }
    }
    __builtin_amdgcn_fence(__ATOMIC_SEQ_CST, "wavefront");
    call = rfl(callv) + 1;
  }
  const unsigned par = (call & 1) * BUF_BYTES;
  const unsigned cvo = par + (is_w ? OPER_BYTES : 0u) + unit * 1024u + lane * 16u;
  const rsrc_t own = mk_rsrc(me + CTRL_BYTES, 2u * BUF_BYTES);

  char* lpeer = me;
#pragma unroll
  for (int i = 0; i < 7; ++i) lpeer = lane == (unsigned)i ? p.peer[i] : lpeer;
  rsrc_t pr[7];
  unsigned pidx[7];
#pragma unroll
  for (int i = 0; i < 7; ++i) {
    pidx[i] = prot + i >= 7 ? prot + i - 7 : prot + i;
    pr[i] = mk_rsrc(pptr[i] + CTRL_BYTES, 2u * BUF_BYTES);
  }
  auto arrive_rdy0 = [&]() {
    unsigned t = 0;
    if (lane == 0) t = __hip_atomic_fetch_add((lu32*)&ctl[2], 1u, __ATOMIC_RELAXED, __HIP_MEMORY_SCOPE_WORKGROUP);
    if (rfl(t) == 7 && lane < 8)
      __hip_atomic_store((gu32*)(lpeer + OFF_RDY0 + ((lane == 7 ? 7u : 6u - lane) * 32u + blockIdx.x) * 128u), call,
                         __ATOMIC_RELAXED, __HIP_MEMORY_SCOPE_SYSTEM);
  };
  auto arrive_own = [&]() {
    unsigned last = 0;
    if (lane == 0 && __hip_atomic_fetch_add((lu32*)&ctl[1], 1u, __ATOMIC_RELAXED, __HIP_MEMORY_SCOPE_WORKGROUP) == 7) {
      __hip_atomic_store(callp, call, __ATOMIC_RELAXED, __HIP_MEMORY_SCOPE_AGENT);
      if ((__hip_atomic_fetch_add((gu32*)(me + OFF_OWN + (blockIdx.x & 7) * 128u), 1u, __ATOMIC_RELAXED, __HIP_MEMORY_SCOPE_AGENT) & 31) == 31)
        last = (__hip_atomic_fetch_add((gu32*)(me + OFF_TOP), 1u, __ATOMIC_RELAXED, __HIP_MEMORY_SCOPE_AGENT) & 7) == 7;
    }
    if (rfl(last) && lane < 8)
      __hip_atomic_store((gu32*)(lpeer + (lane == 7 ? OFF_GO : OFF_RDY + (6u - lane) * 4u)), call, __ATOMIC_RELAXED, __HIP_MEMORY_SCOPE_SYSTEM);
  };
  {
    u4 d[4];
#pragma unroll
    for (int q = 0; q < 4; ++q) d[q] = *(__attribute__((address_space(3))) u4*)(cvl + q * CV_Q + lane * 16);
    if (kb == 1) {
      unsigned long long tq[4];
      TICK(tks);
#pragma unroll
      for (int q = 0; q < NPUSH; ++q) {
#pragma unroll
        for (int i = 0; i < 7; ++i) {
          __builtin_amdgcn_raw_buffer_store_b128(d[q], pr[i], cvo + (COL_A + 7u * q + 6u - pidx[i]) * COL_BYTES, 0, 17);
          asm volatile("s_waitcnt vmcnt(%0)" :: "i"(PACE) : "memory");
          if (q * 7 + i + 1 == NPUSH * 7 - EARLY) arrive_rdy0();
        }
        __builtin_amdgcn_raw_buffer_store_b128(d[q], own, cvo + (4 + q) * COL_BYTES, 0, 17);
      }
#pragma unroll
      for (int q = NPUSH; q < 4; ++q) __builtin_amdgcn_raw_buffer_store_b128(d[q], own, cvo + (4 + q) * COL_BYTES, 0, 17);
      TICK(tk4);
      asm volatile("s_waitcnt vmcnt(0)" ::: "memory");
      TICK(tk5);
      unsigned last = 0;
      if (lane == 0 && __hip_atomic_fetch_add((lu32*)&ctl[3], 1u, __ATOMIC_RELAXED, __HIP_MEMORY_SCOPE_WORKGROUP) == 3)
        last = (__hip_atomic_fetch_add((gu32*)(me + OFF_CNT1), 1u, __ATOMIC_RELAXED, __HIP_MEMORY_SCOPE_AGENT) & 31) == 31;
      if (rfl(last) && lane < 7)
        __hip_atomic_store((gu32*)(lpeer + OFF_PSH + (6u - lane) * 4u), call, __ATOMIC_RELAXED, __HIP_MEMORY_SCOPE_SYSTEM);
      arrive_own();
      TICK(tk6);
      PUT(120, tk0); PUT(121, tk1); PUT(122, tk3); PUT(123, tks); PUT(124, tq[0]); PUT(125, tq[1]); PUT(126, tk5); PUT(127, tq[2]);
    } else {
#pragma unroll
      for (int q = 0; q < 4; ++q) __builtin_amdgcn_raw_buffer_store_b128(d[q], own, cvo + (kb * 4 + q) * COL_BYTES, 0, 17);
      asm volatile("s_waitcnt vmcnt(0)" ::: "memory");
      if (kb == 0) arrive_rdy0();
      arrive_own();
    }
  }
  const rsrc_t ctr = mk_rsrc(me, 0x30000u);

  if (blockIdx.x < NCOMM) {
    const unsigned pv = wid - 4;
    if (!(blockIdx.x < PWG && (pv < PWV || (pv == PWV && blockIdx.x < PXTRA)))) return;
    const unsigned pw = pv * PWG + blockIdx.x;
    gu32* queue = (gu32*)(me + OFF_QUEUE);
    const unsigned qbase = (call - 1) * (unsigned)NT - 3u * NP;
    ((__attribute__((address_space(3))) u4*)(lds + LDS_TAB + wid * 1024u))[lane] = ((const u4*)c_order.t)[lane];
    auto lookup = [&](unsigned n) -> unsigned {
      const unsigned e = rfl(*(__attribute__((address_space(3))) unsigned short*)(lds + LDS_TAB + wid * 1024u + 2 * min(n >> 4, 511u)));
      const unsigned i = n & 15, b = e & 7;
      const unsigned u = b < 4 ? b * 16 + i : 64 + (b - 4) * 4 + (i >> 2) * 16 + (i & 3);
      return n >= (unsigned)NT ? 0xffffu : (e >> 3) * 128 + u;
    };
    bool ok = spin_until(me, call, lane, [&]() {
      unsigned r = 0;
#pragma unroll
      for (int k = 0; k < 4; ++k)
        r |= __builtin_amdgcn_raw_buffer_load_b32(ctr, OFF_RDY0 + k * 8192u + lane * 128u, 0, 17) - call;
      return __all((int)r >= 0); });
    STAMPW(3, 4);
    const unsigned lbase = par + lane * 16u;
    auto src_off = [&](unsigned task) {
      return lbase + (task & 64) * (OPER_BYTES / 64) + (task & 63) * 1024u
             + ((task >> 7) + (task >= 512 ? NPUSH : 0)) * COL_BYTES + (task == 0xffff ? 0x80000000u : 0u);
    };
    const unsigned roff = OFF_RDY + min(lane, 6u) * 4u;
    const unsigned one = 1;
    unsigned prev = 0xffff, cur = ok ? lookup(pw) : 0xffffu;
    bool ready = false;
    u4 v[7];
    {
      const unsigned o = src_off(cur);
#pragma unroll
      for (int i = 0; i < 7; ++i)
        asm volatile("buffer_load_dwordx4 %0, %1, %2, 0 offen sc0 sc1" : "=v"(v[i]) : "v"(o), "s"(pr[i]) : "memory");
    }
    asm volatile("s_waitcnt vmcnt(0)" ::: "memory");
    unsigned it = 0;
    auto round = [&](unsigned& tk, unsigned& rv) {
      const unsigned od = lbase + (cur & 64) * (OPER_BYTES / 64) + (cur & 63) * 1024u + (COL_B + (cur >> 7) * 7u) * COL_BYTES;
      asm volatile("s_waitcnt vmcnt(15)" : "+v"(tk), "+v"(rv) :: "memory");
      unsigned nxt = lookup(rfl(tk) - qbase);
      if (!ready) {
        ready = __all((int)(rv - call) >= 0);
        if (!ready && nxt != 0xffff && nxt >= 4 * 128) {
          ok = spin_until(me, call, lane, [&]() {
            return __all((int)(__builtin_amdgcn_raw_buffer_load_b32(ctr, roff, 0, 17) - call) >= 0); });
          ready = true;
          if (!ok) nxt = 0xffff;
        }
      }
      const unsigned os = src_off(nxt);
#pragma unroll
      for (int i = 0; i < 7; ++i) {
        if (i) asm volatile("s_waitcnt vmcnt(15)" ::: "memory");
        asm volatile("buffer_store_dwordx4 %0, %1, %2, %3 offen sc1\n\tbuffer_load_dwordx4 %0, %4, %5, 0 offen sc0 sc1"
                     : "+v"(v[i]) : "v"(od), "s"(own), "s"(pidx[i] * COL_BYTES), "v"(os), "s"(pr[i]) : "memory");
      }
      const unsigned fo = prev == 0xffff ? OFF_DUMMY + pw * 64u : OFF_FB + (prev & 127) * FB_U + (prev >> 7) * 4u;
      if (lane == 0) {
        asm volatile("buffer_store_dword %0, %1, %2, 0 offen sc1" :: "v"(call), "v"(fo), "s"(ctr) : "memory");
        asm volatile("global_atomic_add %0, %1, %2, off sc0" : "+v"(tk) : "v"(queue), "v"(one) : "memory");
      }
      asm volatile("buffer_load_dword %0, %1, %2, 0 offen sc0 sc1" : "+v"(rv) : "v"(roff), "s"(ctr) : "memory");
      prev = cur; cur = nxt;
    };
    unsigned tka = qbase + NP + pw, tkb = qbase + 2 * NP + pw, rva = call - 1, rvb = call - 1;
    while (cur != 0xffff) {
      round(tka, rva);
      if (cur == 0xffff) break;
      round(tkb, rvb);
    }
    asm volatile("s_waitcnt vmcnt(0)" ::: "memory");
    if (lane == 0 && prev != 0xffff)
      __hip_atomic_store((gu32*)(me + OFF_FB + (prev & 127) * FB_U + (prev >> 7) * 4u), call, __ATOMIC_RELAXED, __HIP_MEMORY_SCOPE_AGENT);
    STAMPW(1, 4);
    return;
  }

  const unsigned gi = blockIdx.x - NCOMM;
  const unsigned xcd = gi & 7, lin = (xcd & 1) * 28 + (gi >> 3);
  const unsigned ti = lin >> 2, tc = (xcd >> 1) * 4 + (lin & 3);
  const unsigned row0 = 16u * (18u * ti + (ti > 5) + (ti > 6) + (ti > 12)), col0 = tc * TN;
  const bool tall = ti == 5 || ti == 6 || ti >= 12;
  STAMP(1);

  const rsrc_t stg = mk_rsrc(me + CTRL_BYTES + par, BUF_BYTES);
  unsigned dvo[5];
#pragma unroll
  for (int i = 0; i < 5; ++i) {
    const unsigned pc = min(wid * 5 + i, 34u);
    if (pc < 19) {
      const unsigned e = pc * 64 + lane, q = e / TM, r = e - q * TM;
      dvo[i] = q * COL_BYTES + (row0 + r) * 16u;
    } else {
      const unsigned e = (pc - 19) * 64 + lane, q = e >> 8, r = e & 255;
      dvo[i] = OPER_BYTES + q * COL_BYTES + (col0 + r) * 16u;
    }
  }
  const unsigned lds0 = (unsigned)(size_t)lds + wid * 5120u;
  auto dma = [&](unsigned s) {
    const unsigned m0 = lds0 + (s & 3) * STAGE;
    const unsigned so = s * (4u * COL_BYTES);
#pragma unroll
    for (int i = 0; i < 5; ++i)
      asm volatile("s_mov_b32 m0, %0\n\ts_nop 0\n\tbuffer_load_dwordx4 %1, %2, %3 offen sc1 lds"
                   :: "s"(m0 + i * 1024u), "v"(dvo[i]), "s"(stg), "s"(so) : "memory");
  };

  unsigned poff;
  {
    const unsigned wk = min(lane / 10, 5u), wh = lane % 10;
    const unsigned u = wh < 6 ? min((row0 >> 6) + wh, (row0 + TM - 1) >> 6) : 64 + tc * 4 + (wh - 6);
    poff = OFF_FB + u * FB_U + wk * 4u;
  }
  unsigned kg0 = 0;
  unsigned avail = COL_A;
  bool ok = true;
  auto poll_issue = [&]() -> unsigned { return __builtin_amdgcn_raw_buffer_load_b32(ctr, poff, kg0 * 4u, 16); };
  auto poll_eval = [&](unsigned v) {
    const unsigned long long m = __ballot((int)(v - call) >= 0) & 0x0fffffffffffffffull;
    kg0 += (unsigned)__builtin_ctzll(~m) / 10u;
    avail = COL_B + kg0 * 7;
  };

  const unsigned wr = wid >> 2, wc = wid & 3;
  const unsigned a_off = (lane >> 4) * A_COL + (wr * 160 + (lane & 15)) * 16;
  const unsigned b_off = A_BYTES + (lane >> 4) * B_COL + (wc * 64 + (lane & 15)) * 16;
  f4 acc[10][4];
#pragma unroll
  for (int m = 0; m < 10; ++m)
#pragma unroll
    for (int n = 0; n < 4; ++n) acc[m][n] = f4{0.f, 0.f, 0.f, 0.f};

  unsigned pv = 0, pend = 0;
  if (wid == 7)
    ok = spin_until(me, call, lane, [&]() { return (int)(rfl(__builtin_amdgcn_raw_buffer_load_b32(ctr, OFF_GO, 0, 17)) - call) >= 0; });
  asm volatile("s_barrier" ::: "memory");
  STAMP(2);
  STAMPC(5);
  if (wid != 7) {
#pragma unroll
    for (int s = 0; s < LOOK; ++s) dma(s);
    asm volatile("s_waitcnt vmcnt(%0)" :: "i"(5 * (LOOK - 1)) : "memory");
  }
  asm volatile("s_barrier" ::: "memory");
  if (wr == 1) asm volatile("s_barrier" ::: "memory");
  for (unsigned j = 0; j < 128; ++j) {
    lchar* sb = lds + (j & 3) * STAGE;
    h8 a[10], b[4];
#pragma unroll
    for (int n = 0; n < 4; ++n) b[n] = *(__attribute__((address_space(3))) h8*)(sb + b_off + n * 256);
#pragma unroll
    for (int m = 0; m < 10; ++m) a[m] = *(__attribute__((address_space(3))) h8*)(sb + a_off + m * 256);
    __builtin_amdgcn_sched_barrier(0);
    if (wid != 7) {
      if (j + LOOK < 128) {
        dma(j + LOOK);
        asm volatile("s_waitcnt vmcnt(%0)" :: "i"(5 * (LOOK - 1)) : "memory");
      } else if (LOOK == 3 && j == 125) {
        asm volatile("s_waitcnt vmcnt(5)" ::: "memory");
      } else {
        asm volatile("s_waitcnt vmcnt(0)" ::: "memory");
      }
    } else if (j + LOOK + 1 < 128) {
      const unsigned need = (j + LOOK + 2) * 4;
      if (ok && avail < 512) {
        if (NPUSH && avail == COL_A) {
          if (need > COL_A) {
            ok = spin_until(me, call, lane, [&]() {
              return __all((int)(__builtin_amdgcn_raw_buffer_load_b32(ctr, OFF_PSH + min(lane, 6u) * 4u, 0, 17) - call) >= 0); });
            avail = COL_B;
          }
        } else {
          if (pend && (avail < need || pend > 2)) { poll_eval(pv); pend = 0; }
          if (avail < need) ok = spin_until(me, call, lane, [&]() { poll_eval(poll_issue()); return avail >= need; });
          if (pend) ++pend;
          else if (avail < need + 16) { pv = poll_issue(); pend = 1; }
        }
      }
    } else if (j + LOOK + 1 == 128) {
      if (lane == 0) ctl[0] = ok ? 0u : 1u;
    }
    asm volatile("s_waitcnt lgkmcnt(0)\n\ts_barrier" ::: "memory");
    __builtin_amdgcn_sched_barrier(0);
    __builtin_amdgcn_s_setprio(1);
#pragma unroll
    for (int m = 0; m < 9; ++m)
#pragma unroll
      for (int n = 0; n < 4; ++n) acc[m][n] = __builtin_amdgcn_mfma_f32_16x16x32_f16(a[m], b[n], acc[m][n], 0, 0, 0);
    if (wr == 0) {
#pragma unroll
      for (int n = 0; n < 4; ++n) acc[9][n] = __builtin_amdgcn_mfma_f32_16x16x32_f16(a[9], b[n], acc[9][n], 0, 0, 0);
    }
    __builtin_amdgcn_s_setprio(0);
    __builtin_amdgcn_sched_barrier(0);
    if (wr == 0 || j < 127) asm volatile("s_barrier" ::: "memory");
  }
  STAMP(3);
  STAMPC(6);
  const bool bad = rfl(ctl[0]) != 0;
  const rsrc_t ro = mk_rsrc(outp, OUT_BYTES);
  const unsigned ovo = ((row0 + wr * 160 + 4 * (lane >> 4)) * 4096u + col0 + wc * 64 + 4 * (lane & 15)) * 4u;
  const int mcnt = wr == 0 ? 10 : tall ? 9 : 8;
#pragma unroll
  for (int m = 0; m < 10; ++m) {
    if (m < mcnt) {
#pragma unroll
      for (int j = 0; j < 4; ++j) {
        f4 o = f4{acc[m][0][j], acc[m][1][j], acc[m][2][j], acc[m][3][j]};
        if (bad) o = f4{__builtin_nanf(""), __builtin_nanf(""), __builtin_nanf(""), __builtin_nanf("")};
        __builtin_amdgcn_raw_buffer_store_b128(__builtin_bit_cast(u4, o), ro, ovo + (16 * m + j) * 16384u, 0, OAUX);
      }
    }
  }
}

}

extern "C" void kernel_launch(int rank, int nranks, ncclComm_t comm,
                              hipStream_t stream,
                              void* const* d_in, const int* in_sizes, int n_in,
                              void* d_out, int out_size,
                              const int* in_dtypes, int out_dtype,
                              void* const* d_out_all,
                              void* const* d_ws_all, size_t ws_bytes) {
  if (nranks != NR || (unsigned)rank >= (unsigned)NR || ws_bytes < WS_NEED || n_in != 2 ||
      in_sizes[0] != 4096 * 512 || in_sizes[1] != 4096 * 512 || out_size != 4096 * 4096) {
    fprintf(stderr, "kernel_launch: unexpected problem shape\n");
    return;
  }
  Params p;
  p.x = (const float*)d_in[0];
  p.w = (const float*)d_in[1];
  p.out = (float*)d_out;
  p.me = (char*)d_ws_all[rank];
  for (int i = 0; i < 7; ++i) p.peer[i] = (char*)d_ws_all[(rank + 1 + i) % NR];
  hipLaunchKernelGGL(gemm_ar_kernel, dim3(NWG), dim3(512), 0, stream, p);
  HIP_CHECK(hipGetLastError());
}
```

```cpp
#include <hip/hip_runtime.h>
#include <rccl/rccl.h>
#include <cstdio>

#define HIP_CHECK(expr) do { hipError_t _e = (expr); if (_e != hipSuccess) { \
    fprintf(stderr, "HIP error %s at %s:%d\n", hipGetErrorString(_e), __FILE__, __LINE__); \
    return; } } while (0)

#define OAUX 16
#define SLEEP 2
#define JSW 96
#define TCMAP 3
#define TWIN 2
#define PWG 32
#define PWV 2
#define NPUSH 3
#define EARLY 10
#define PACE 4
#define GDELAY0 30
#define GDELAY 30
#define PXTRA 0

namespace {

typedef _Float16 h8 __attribute__((ext_vector_type(8)));
typedef _Float16 h4 __attribute__((ext_vector_type(4)));
typedef float f4 __attribute__((ext_vector_type(4)));
typedef unsigned u4 __attribute__((ext_vector_type(4)));
typedef unsigned u2 __attribute__((ext_vector_type(2)));
typedef __attribute__((address_space(1))) unsigned gu32;
typedef __attribute__((address_space(3))) char lchar;
typedef __attribute__((address_space(3))) unsigned lu32;
typedef __amdgpu_buffer_rsrc_t rsrc_t;

constexpr int NR = 8;
constexpr int NCOMM = 32, NGEMM = 224, NWG = NCOMM + NGEMM;
constexpr unsigned SHARD_BYTES = 4096u * 512u * 4u;
constexpr unsigned OUT_BYTES = 4096u * 4096u * 4u;
constexpr unsigned COL_BYTES = 4096u * 16u;
constexpr unsigned OPER_BYTES = 512u * COL_BYTES;
constexpr unsigned BUF_BYTES = 2u * OPER_BYTES;
constexpr size_t CTRL_BYTES = 1u << 20;
constexpr size_t WS_NEED = CTRL_BYTES + 2ull * BUF_BYTES;
constexpr unsigned COL_A = 64, COL_B = 64 + 7 * NPUSH;
constexpr unsigned OFF_CALL = 0;
constexpr unsigned OFF_TMO = 0x8000;
constexpr unsigned OFF_QUEUE = 0x8100;
constexpr unsigned OFF_RDY = 0x8200;
constexpr unsigned OFF_RDY0 = 0x20000;
constexpr unsigned OFF_TOP = 0x8500;
constexpr unsigned OFF_GO = 0x8700;
constexpr unsigned OFF_PSH = 0x8400;
constexpr unsigned OFF_CNT1 = 0x8600;
constexpr unsigned OFF_OWN = 0x9000;
constexpr unsigned OFF_DUMMY = 0xC000;
constexpr unsigned OFF_FB = 0x10000;
constexpr unsigned FB_U = 384;
constexpr unsigned long long DEADLINE = 30000000ull;

constexpr int TM = 304, TN = 256;
constexpr unsigned A_COL = TM * 16, B_COL = TN * 16;
constexpr unsigned A_BYTES = 4 * A_COL, B_BYTES = 4 * B_COL, STAGE = A_BYTES + B_BYTES;
constexpr unsigned NMAIN = 121, COL_T = NMAIN * 4;
constexpr unsigned TW_BYTES = 28 * B_COL, TX_BYTES = 28 * 512;
constexpr unsigned LDS_CTL = TW_BYTES + 3 * TX_BYTES;
constexpr unsigned LDS_BYTES = LDS_CTL + 64;
static_assert(LDS_CTL >= 4 * STAGE && LDS_BYTES <= 160 * 1024, "LDS layout");
constexpr unsigned CV_Q = 1024 + 32, CV_WAVE = 4 * CV_Q;

constexpr int NKG = 64 - NPUSH, NT = NKG * 128;
constexpr int NP = PWG * PWV + PXTRA;

template <int V> struct IC { static constexpr int value = V; };

struct Params {
  const float* x;
  const float* w;
  float* out;
  char* me;
  char* peer[7];
};

__device__ __forceinline__ rsrc_t mk_rsrc(const void* p, unsigned bytes) {
  return __builtin_amdgcn_make_buffer_rsrc((void*)p, (short)0, (int)bytes, 0x00020000);
}
__device__ __forceinline__ unsigned rfl(unsigned v) { return __builtin_amdgcn_readfirstlane(v); }

#define STAMPW(i, w) do {} while (0)
#define STAMPC(i) do {} while (0)
#define STAMP(i) STAMPW(i, 7)
#define TICK(x)
#define PUT(i, x)

template <class F>
__device__ __forceinline__ bool spin_until(char* me, unsigned call, unsigned lane, F cond) {
  gu32* tmo = (gu32*)(me + OFF_TMO);
  const unsigned long long t0 = __builtin_amdgcn_s_memrealtime();
  for (unsigned spin = 1;; ++spin) {
    if (cond()) return true;
    if ((spin & 63) == 0 && (__builtin_amdgcn_s_memrealtime() - t0 > DEADLINE ||
        rfl(__hip_atomic_load(tmo, __ATOMIC_RELAXED, __HIP_MEMORY_SCOPE_AGENT)) == call)) {
      if (lane == 0) __hip_atomic_store(tmo, call, __ATOMIC_RELAXED, __HIP_MEMORY_SCOPE_AGENT);
      return false;
    }
    __builtin_amdgcn_s_sleep(SLEEP);
  }
}

__global__ void __launch_bounds__(512, 2) gemm_ar_kernel(Params p) {
  __shared__ __attribute__((aligned(16))) char smem[LDS_BYTES];
  TICK(tk0);
  lchar* lds = (lchar*)smem;
  char* const me = p.me;
  float* const outp = p.out;
  const float* const xp = p.x;
  const float* const wp = p.w;
  const unsigned lane = threadIdx.x & 63;
  const unsigned wid = rfl(threadIdx.x >> 6);
  const unsigned prot = (blockIdx.x * 4 + wid) % 7;
  char* pptr[7];
  gu32* callp = (gu32*)(me + OFF_CALL + blockIdx.x * 128u);
  volatile lu32* ctl = (volatile lu32*)(lds + LDS_CTL);
  STAMP(0);
  const bool is_comm = blockIdx.x < NCOMM;
  const unsigned gi = blockIdx.x - NCOMM;
  const unsigned ctask = is_comm ? (wid >> 2) * (15u * 128u) + blockIdx.x * 4 + (wid & 3)
                       : wid >= 4 ? 256u + 768u + (wid - 4) * NGEMM + gi
                       : gi < 32 ? 128u + gi * 4 + wid : 256u + wid * 192u + (gi - 32);
  const bool urgent = ctask < 256;
  lchar* cvl = lds + wid * CV_WAVE;
  f4 v[8];
  auto cv_load = [&]() __attribute__((always_inline)) {
    const unsigned kb = ctask >> 7, unit = ctask & 63;
    const bool is_w = (ctask & 64) != 0;
    const rsrc_t src = mk_rsrc(is_w ? wp : xp, SHARD_BYTES);
    const unsigned voff = is_w ? (kb * 32u + (lane >> 4) * 8u) * 16384u + unit * 256u + (lane & 15) * 16u
                               : (unit * 64u + (lane >> 3)) * 2048u + kb * 128u + (lane & 7) * 16u;
#pragma unroll
    for (int i = 0; i < 8; ++i)
      v[i] = __builtin_bit_cast(f4, __builtin_amdgcn_raw_buffer_load_b128(src, voff, i * 16384u, 0));
  };
  if (!urgent) {
    if (threadIdx.x == 448) { ctl[1] = 0; ctl[2] = 0; ctl[3] = 0; }
    asm volatile("s_waitcnt lgkmcnt(0)\n\ts_barrier" ::: "memory");
    __builtin_amdgcn_s_sleep(GDELAY0);
    if (!is_comm)
      for (unsigned k = 3; k < (ctask >> 7); k += 2) __builtin_amdgcn_s_sleep(GDELAY);
  }
  cv_load();
  const unsigned callv = __hip_atomic_load(callp, __ATOMIC_RELAXED, __HIP_MEMORY_SCOPE_AGENT);
  if (urgent) asm volatile("s_barrier" ::: "memory");
#pragma unroll
  for (int i = 0; i < 7; ++i) pptr[i] = p.peer[prot + i >= 7 ? prot + i - 7 : prot + i];
  char* lpeer = me;
#pragma unroll
  for (int i = 0; i < 7; ++i) lpeer = lane == (unsigned)i ? p.peer[i] : lpeer;
  rsrc_t pr[7];
  unsigned pidx[7];
#pragma unroll
  for (int i = 0; i < 7; ++i) {
    pidx[i] = prot + i >= 7 ? prot + i - 7 : prot + i;
    pr[i] = mk_rsrc(pptr[i] + CTRL_BYTES, 2u * BUF_BYTES);
  }
  const rsrc_t own = mk_rsrc(me + CTRL_BYTES, 2u * BUF_BYTES);
  const unsigned call = rfl(callv) + 1;
  const unsigned par = (call & 1) * BUF_BYTES;
  auto arrive_rdy0 = [&]() {
    unsigned t = 0;
    if (lane == 0) t = __hip_atomic_fetch_add((lu32*)&ctl[2], 1u, __ATOMIC_RELAXED, __HIP_MEMORY_SCOPE_WORKGROUP);
    if (rfl(t) == 3 && lane < 8)
      __hip_atomic_store((gu32*)(lpeer + OFF_RDY0 + ((lane == 7 ? 7u : 6u - lane) * 32u + (blockIdx.x & 31)) * 128u + (is_comm ? 0u : 4u)),
                         call, __ATOMIC_RELAXED, __HIP_MEMORY_SCOPE_SYSTEM);
  };
  {
    const unsigned kb = ctask >> 7;
    if (ctask & 64) {
#pragma unroll
      for (int u = 0; u < 4; ++u) {
        h8 g;
#pragma unroll
        for (int i = 0; i < 8; ++i) g[i] = (_Float16)v[i][u];
        *(__attribute__((address_space(3))) h8*)(cvl + (lane >> 4) * CV_Q + (16 * u + (lane & 15)) * 16) = g;
      }
    } else {
#pragma unroll
      for (int i = 0; i < 8; ++i) {
        h4 g;
#pragma unroll
        for (int u = 0; u < 4; ++u) g[u] = (_Float16)v[i][u];
        *(__attribute__((address_space(3))) h4*)(cvl + ((lane & 7) >> 1) * CV_Q + (8 * i + (lane >> 3)) * 16 + (lane & 1) * 8) = g;
      }
    }
    __builtin_amdgcn_fence(__ATOMIC_SEQ_CST, "wavefront");
    const unsigned cvo = par + (ctask & 64) * (OPER_BYTES / 64) + (ctask & 63) * 1024u + lane * 16u;
    u4 d[4];
#pragma unroll
    for (int q = 0; q < 4; ++q) d[q] = *(__attribute__((address_space(3))) u4*)(cvl + q * CV_Q + lane * 16);
    if (kb == 1) {
      TICK(tks);
#pragma unroll
      for (int q = 0; q < NPUSH; ++q) {
#pragma unroll
        for (int i = 0; i < 7; ++i) {
          __builtin_amdgcn_raw_buffer_store_b128(d[q], pr[i], cvo + (COL_A + 7u * q + 6u - pidx[i]) * COL_BYTES, 0, 17);
          asm volatile("s_waitcnt vmcnt(%0)" :: "i"(PACE) : "memory");
          if (q * 7 + i + 1 == NPUSH * 7 - EARLY) arrive_rdy0();
        }
      }
      TICK(tk4);
      PUT(120, tk0); PUT(123, tks); PUT(124, tk4);
    }
#pragma unroll
    for (int q = 0; q < 4; ++q) __builtin_amdgcn_raw_buffer_store_b128(d[q], own, cvo + (kb * 4 + q) * COL_BYTES, 0, 17);
    asm volatile("s_waitcnt vmcnt(0)" ::: "memory");
    if (kb == 1) {
      TICK(tk5);
      PUT(126, tk5);
      unsigned last = 0;
      if (lane == 0 && __hip_atomic_fetch_add((lu32*)&ctl[3], 1u, __ATOMIC_RELAXED, __HIP_MEMORY_SCOPE_WORKGROUP) == 3)
        last = (__hip_atomic_fetch_add((gu32*)(me + OFF_CNT1), 1u, __ATOMIC_RELAXED, __HIP_MEMORY_SCOPE_AGENT) & 31) == 31;
      if (rfl(last) && lane < 7)
        __hip_atomic_store((gu32*)(lpeer + OFF_PSH + (6u - lane) * 4u), call, __ATOMIC_RELAXED, __HIP_MEMORY_SCOPE_SYSTEM);
    }
    if (kb == 0) arrive_rdy0();
    unsigned last = 0;
    if (lane == 0 && __hip_atomic_fetch_add((lu32*)&ctl[1], 1u, __ATOMIC_RELAXED, __HIP_MEMORY_SCOPE_WORKGROUP) == 7u) {
      __hip_atomic_store(callp, call, __ATOMIC_RELAXED, __HIP_MEMORY_SCOPE_AGENT);
      if ((__hip_atomic_fetch_add((gu32*)(me + OFF_OWN + (blockIdx.x & 7) * 128u), 1u, __ATOMIC_RELAXED, __HIP_MEMORY_SCOPE_AGENT) & 31) == 31)
        last = (__hip_atomic_fetch_add((gu32*)(me + OFF_TOP), 1u, __ATOMIC_RELAXED, __HIP_MEMORY_SCOPE_AGENT) & 7) == 7;
    }
    if (rfl(last) && lane < 8)
      __hip_atomic_store((gu32*)(lpeer + (lane == 7 ? OFF_GO : OFF_RDY + (6u - lane) * 4u)), call, __ATOMIC_RELAXED, __HIP_MEMORY_SCOPE_SYSTEM);
  }
  const rsrc_t ctr = mk_rsrc(me, 0x30000u);

  if (blockIdx.x < NCOMM) {
    const unsigned pv = wid;
    if (!(blockIdx.x < PWG && (pv < PWV || (pv == PWV && blockIdx.x < PXTRA)))) return;
    const unsigned pw = pv * PWG + blockIdx.x;
    gu32* queue = (gu32*)(me + OFF_QUEUE);
    const unsigned qbase = (call - 1) * (unsigned)NT - 3u * NP;
    auto lookup = [&](unsigned n) -> unsigned {
      const unsigned i = n - (NT - 512u);
      const unsigned t = i < 256 ? (NKG - 4 + (i >> 6)) * 128 + 64 + (i & 63) : (NKG - 4 + (i & 3)) * 128 + ((i - 256) >> 2);
      return n >= (unsigned)NT ? 0xffffu : n < NT - 512u ? n : t;
    };
    bool ok = spin_until(me, call, lane, [&]() {
      unsigned r = 0;
#pragma unroll
      for (int k = 0; k < 4; ++k) {
        const u2 f = __builtin_amdgcn_raw_buffer_load_b64(ctr, OFF_RDY0 + k * 8192u + lane * 128u, 0, 17);
        r |= (f[0] - call) | (f[1] - call);
      }
      return __all((int)r >= 0); });
    STAMPW(3, 0);
    const unsigned lbase = par + lane * 16u;
    auto src_off = [&](unsigned task) {
      return lbase + (task & 64) * (OPER_BYTES / 64) + (task & 63) * 1024u
             + ((task >> 7) + (task >= 512 ? NPUSH : 0)) * COL_BYTES + (task == 0xffff ? 0x80000000u : 0u);
    };
    const unsigned roff = OFF_RDY + min(lane, 6u) * 4u;
    const unsigned one = 1;
    unsigned prev = 0xffff, cur = ok ? lookup(pw) : 0xffffu;
    bool ready = false;
    u4 v[7];
    {
      const unsigned o = src_off(cur);
#pragma unroll
      for (int i = 0; i < 7; ++i)
        asm volatile("buffer_load_dwordx4 %0, %1, %2, 0 offen sc0 sc1" : "=v"(v[i]) : "v"(o), "s"(pr[i]) : "memory");
    }
    asm volatile("s_waitcnt vmcnt(0)" ::: "memory");
    unsigned it = 0;
    auto round = [&](unsigned& tk, unsigned& rv) {
      const unsigned od = lbase + (cur & 64) * (OPER_BYTES / 64) + (cur & 63) * 1024u + (COL_B + (cur >> 7) * 7u) * COL_BYTES;
      asm volatile("s_waitcnt vmcnt(15)" : "+v"(tk), "+v"(rv) :: "memory");
      unsigned nxt = lookup(rfl(tk) - qbase);
      if (!ready) {
        ready = __all((int)(rv - call) >= 0);
        if (!ready && nxt != 0xffff && nxt >= 4 * 128) {
          ok = spin_until(me, call, lane, [&]() {
            return __all((int)(__builtin_amdgcn_raw_buffer_load_b32(ctr, roff, 0, 17) - call) >= 0); });
          ready = true;
          if (!ok) nxt = 0xffff;
        }
      }
      const unsigned os = src_off(nxt);
#pragma unroll
      for (int i = 0; i < 7; ++i) {
        if (i) asm volatile("s_waitcnt vmcnt(15)" ::: "memory");
        asm volatile("buffer_store_dwordx4 %0, %1, %2, %3 offen sc1\n\tbuffer_load_dwordx4 %0, %4, %5, 0 offen sc0 sc1"
                     : "+v"(v[i]) : "v"(od), "s"(own), "s"(pidx[i] * COL_BYTES), "v"(os), "s"(pr[i]) : "memory");
      }
      const unsigned fo = prev == 0xffff ? OFF_DUMMY + pw * 64u : OFF_FB + (prev & 127) * FB_U + (prev >> 7) * 4u;
      if (lane == 0) {
        asm volatile("buffer_store_dword %0, %1, %2, 0 offen sc1" :: "v"(call), "v"(fo), "s"(ctr) : "memory");
        asm volatile("global_atomic_add %0, %1, %2, off sc0" : "+v"(tk) : "v"(queue), "v"(one) : "memory");
      }
      asm volatile("buffer_load_dword %0, %1, %2, 0 offen sc0 sc1" : "+v"(rv) : "v"(roff), "s"(ctr) : "memory");
      prev = cur; cur = nxt;
    };
    unsigned tka = qbase + NP + pw, tkb = qbase + 2 * NP + pw, rva = call - 1, rvb = call - 1;
    while (cur != 0xffff) {
      round(tka, rva);
      if (cur == 0xffff) break;
      round(tkb, rvb);
    }
    asm volatile("s_waitcnt vmcnt(4)" ::: "memory");
    if (lane == 0 && prev != 0xffff)
      __hip_atomic_store((gu32*)(me + OFF_FB + (prev & 127) * FB_U + (prev >> 7) * 4u), call, __ATOMIC_RELAXED, __HIP_MEMORY_SCOPE_AGENT);
    STAMPW(1, 0);
    return;
  }

  const unsigned xcd = gi & 7, ti7 = gi >> 5, ti = (xcd & 1) * 7 + ti7;
  const unsigned tp = xcd >> 1, tq = (gi >> 3) & 3;
  const unsigned tc = TCMAP == 0 ? tp * 4 + tq : TCMAP == 1 ? tq * 4 + tp : TCMAP == 2 ? tp * 2 + (tq & 1) + (tq >> 1) * 8 : tq * 4 + ((tp + tq) & 3), col0 = tc * TN;
  const bool tall = ti7 < 2;
  const unsigned p8 = tall ? 224 + 2 * ((xcd & 1) * 2 + ti7) : 232 + (xcd & 1) * 5 + (ti7 - 2);
  auto blk = [&](unsigned g, unsigned m) -> unsigned {
    return m < 8 ? 28 * m + 2 * ti + g : m == 8 ? p8 + (tall ? g : 0u) : 242 + ti;
  };
  STAMP(1);

  const rsrc_t stg = mk_rsrc(me + CTRL_BYTES + par, BUF_BYTES);
  constexpr int ND0 = 9;
  const unsigned pc0 = min(wid * ND0, 35u - ND0);
  unsigned dvo[ND0];
#pragma unroll
  for (int i = 0; i < ND0; ++i) {
    const unsigned pc = pc0 + i;
    if (pc < 19) {
      const unsigned e = pc * 64 + lane, q = e / TM, r = e - q * TM, g = r >= 160, m = (r - 160 * g) >> 4;
      dvo[i] = q * COL_BYTES + (blk(g, m) * 16u + (r & 15)) * 16u;
    } else {
      const unsigned e = (pc - 19) * 64 + lane, q = e >> 8, r = e & 255;
      dvo[i] = OPER_BYTES + q * COL_BYTES + (col0 + r) * 16u;
    }
  }
  const unsigned ldsb = (unsigned)(size_t)lds;
  auto dma = [&](unsigned m0, unsigned vo, unsigned so) __attribute__((always_inline)) {
    asm volatile("s_mov_b32 m0, %0\n\ts_nop 0\n\tbuffer_load_dwordx4 %1, %2, %3 offen sc1 lds"
                 :: "s"(m0), "v"(vo), "s"(stg), "s"(so) : "memory");
  };

  unsigned poff;
  {
    const unsigned wk = min(lane / 14, 3u), wh = lane % 14;
    const unsigned u = wh < 10 ? blk(0, wh) >> 2 : 64 + tc * 4 + (wh - 10);
    poff = OFF_FB + u * FB_U + wk * 4u;
  }
  unsigned kg0 = 0;
  unsigned avail = COL_A;
  bool ok = true;
  auto poll_issue = [&]() -> unsigned { return __builtin_amdgcn_raw_buffer_load_b32(ctr, poff, kg0 * 4u, 16); };
  auto poll_eval = [&](unsigned v) {
    const unsigned long long m = __ballot((int)(v - call) >= 0) & 0x00ffffffffffffffull;
    kg0 = min(kg0 + (unsigned)__builtin_ctzll(~m) / 14u, (unsigned)NKG - 4);
    avail = COL_B + kg0 * 7;
  };

  const unsigned wr = wid >> 2, wc = wid & 3;
  const unsigned a_off = (lane >> 4) * A_COL + (wr * 160 + (lane & 15)) * 16;
  const unsigned b_off = A_BYTES + (lane >> 4) * B_COL + (wc * 64 + (lane & 15)) * 16;
  f4 acc[10][4];
#pragma unroll
  for (int m = 0; m < 10; ++m)
#pragma unroll
    for (int n = 0; n < 4; ++n) acc[m][n] = f4{0.f, 0.f, 0.f, 0.f};

  unsigned pv = 0, pend = 0;
  if (wid == 7)
    ok = spin_until(me, call, lane, [&]() { return (int)(rfl(__builtin_amdgcn_raw_buffer_load_b32(ctr, OFF_GO, 0, 17)) - call) >= 0; });
  asm volatile("s_barrier" ::: "memory");
  STAMP(2);
  STAMPC(5);
  unsigned jev = COL_A / 4 - 4;
  auto poll_step = [&](unsigned j) __attribute__((always_inline)) {
    const unsigned need = (j + (j < JSW ? 4 : 3)) * 4;
    if (!ok) {
    } else if (avail == COL_A) {
      if (need > COL_A) {
        ok = spin_until(me, call, lane, [&]() {
          return __all((int)(__builtin_amdgcn_raw_buffer_load_b32(ctr, OFF_PSH + min(lane, 6u) * 4u, 0, 17) - call) >= 0); });
        avail = COL_B;
      }
    } else {
      if (pend) poll_eval(pv);
      if (avail < need) ok = spin_until(me, call, lane, [&]() { poll_eval(poll_issue()); return avail >= need; });
    }
    asm volatile("s_waitcnt lgkmcnt(0)" ::: "memory");
    const unsigned jblk = avail / 4 - (avail / 4 < JSW + 3 ? 3 : 2);
    pend = 0;
    if (!ok || avail >= COL_T) {
      jev = 1000;
    } else if (avail != COL_A && jblk <= j + 4) {
      pv = poll_issue(); pend = 1;
      jev = min(jblk, j + 2);
    } else {
      jev = avail == COL_A ? jblk : jblk - 4;
    }
  };
  const unsigned la = ldsb + a_off, lb = ldsb + b_off;
  h8 a[10], b[4];
  auto kloop = [&](auto mu_c, auto role_c) __attribute__((always_inline)) {
    constexpr int MU = decltype(mu_c)::value;
    constexpr int ROLE = decltype(role_c)::value;
#define RD(dst, base, off) asm volatile("ds_read_b128 %0, %1 offset:%2" : "+v"(dst) : "v"(base), "i"(off) : "memory")
#define SB __builtin_amdgcn_sched_barrier(0)
    const unsigned lds0 = ldsb + pc0 * 1024u;
    if (ROLE == 0) {
#pragma unroll
      for (int st = 0; st < 3; ++st)
#pragma unroll
        for (int i = 0; i < ND0; ++i) dma(lds0 + st * STAGE + i * 1024u, dvo[i], st * (4u * COL_BYTES));
      asm volatile("s_waitcnt vmcnt(%0)" :: "i"(2 * ND0) : "memory");
    }
    asm volatile("s_barrier" ::: "memory");
    RD(b[0], lb, 0); RD(b[1], lb, 256);
#pragma unroll
    for (int m = 0; m < MU; ++m) RD(a[m], la, m * 256);
    RD(b[2], lb, 512); RD(b[3], lb, 768);
    if (ROLE != 0) __builtin_amdgcn_s_setprio(1);
    for (unsigned j = 0; j < NMAIN; ++j) {
      if (ROLE == 2 && __builtin_expect(j >= jev, 0)) poll_step(j);
      if (ROLE != 0) asm volatile("s_barrier" ::: "memory");
      const unsigned sd = j < JSW ? j + 3 : j + 2;
      const unsigned m0 = lds0 + (sd & 3) * STAGE;
      const unsigned so = sd * (4u * COL_BYTES);
      const unsigned nb = ((j + 1) & 3) * STAGE;
      const unsigned na = la + nb, nbb = lb + nb;
      SB;
#pragma unroll
      for (int m = 0; m < MU; ++m) {
        if (m == 0) asm volatile("s_waitcnt lgkmcnt(%3)" : "+v"(a[0]), "+v"(b[0]), "+v"(b[1]) : "i"(MU + 1) : "memory");
        else asm volatile("s_waitcnt lgkmcnt(%1)" : "+v"(a[m]) : "i"(MU + 1 - m) : "memory");
        acc[m][0] = __builtin_amdgcn_mfma_f32_16x16x32_f16(a[m], b[0], acc[m][0], 0, 0, 0);
        acc[m][1] = __builtin_amdgcn_mfma_f32_16x16x32_f16(a[m], b[1], acc[m][1], 0, 0, 0);
        SB;
      }
      if (ROLE == 0) {
        if (j > JSW) asm volatile("s_waitcnt vmcnt(0)" ::: "memory");
        asm volatile("s_waitcnt vmcnt(%0)\n\ts_barrier" :: "i"(ND0) : "memory");
      }
      RD(b[0], nbb, 0); RD(b[1], nbb, 256);
      if (ROLE == 0 && j != JSW && sd < NMAIN) {
#pragma unroll
        for (int i = 0; i < ND0; ++i) dma(m0 + i * 1024u, dvo[i], so);
      }
      asm volatile("s_waitcnt lgkmcnt(2)" : "+v"(b[2]), "+v"(b[3]) :: "memory");
      SB;
#pragma unroll
      for (int m = 0; m < MU; ++m) {
        acc[m][2] = __builtin_amdgcn_mfma_f32_16x16x32_f16(a[m], b[2], acc[m][2], 0, 0, 0);
        acc[m][3] = __builtin_amdgcn_mfma_f32_16x16x32_f16(a[m], b[3], acc[m][3], 0, 0, 0);
        SB;
        RD(a[m], na, m * 256);
        SB;
      }
      RD(b[2], nbb, 512); RD(b[3], nbb, 768);
      SB;
    }
    asm volatile("s_waitcnt lgkmcnt(0)" :: "v"(a[0]), "v"(a[1]), "v"(a[2]), "v"(a[3]), "v"(a[4]), "v"(a[5]),
                 "v"(a[6]), "v"(a[7]), "v"(a[8]), "v"(a[MU - 1]), "v"(b[0]), "v"(b[1]), "v"(b[2]), "v"(b[3]) : "memory");
    if (ROLE != 0) __builtin_amdgcn_s_setprio(0);
  };
  if (wid < 4) kloop(IC<10>{}, IC<0>{});
  else if (wid < 7) kloop(IC<9>{}, IC<1>{});
  else kloop(IC<9>{}, IC<2>{});
  STAMP(3);
  STAMPC(6);

  {
    unsigned have = 0;
    auto tail_eval = [&](unsigned v) {
      const unsigned long long m = __ballot((int)(v - call) >= 0);
      const unsigned h3 = (unsigned)(m & (m >> 14) & (m >> 28));
      have = (h3 & (unsigned)(m >> 42) & 0x3fffu) | ((h3 & 0x3c00u) << 4);
    };
    auto tail_wait = [&](unsigned mask) __attribute__((always_inline)) {
      if (__builtin_expect(ok && (have & mask) != mask, 0))
        ok = spin_until(me, call, lane, [&]() {
          tail_eval(__builtin_amdgcn_raw_buffer_load_b32(ctr, poff, (NKG - 4) * 4u, 16)); return (have & mask) == mask; });
    };
    const bool dmaw = wid < 7;
    const unsigned wvo = OPER_BYTES + (col0 + lane) * 16u;
    auto dma_w = [&](unsigned c) __attribute__((always_inline)) {
#pragma unroll
      for (int i = 0; i < 4; ++i) dma(ldsb + c * B_COL + i * 1024u, wvo, (COL_T + c) * COL_BYTES + i * 1024u);
    };
    if (wid == 7) tail_wait(0x3c000u);
    asm volatile("s_barrier" ::: "memory");
    STAMP(8 + 60);
    if (dmaw) { dma_w(wid); dma_w(7 + wid); dma_w(14 + wid); }
    else tail_wait(0x3c01u);
    asm volatile("s_barrier" ::: "memory");
    STAMP(8 + 61);
    const unsigned xg = (lane >> 4) & 1;
    const unsigned xvo = (COL_T + wid * 4 + (lane >> 5)) * COL_BYTES + (lane & 15) * 16u;
    auto xreg = [](int m) -> unsigned {
      return m < 3 ? TW_BYTES + m * TX_BYTES : m == 3 ? TW_BYTES : m == 4 ? 6 * 16384u : m == 5 ? 5 * 16384u
           : m == 6 ? TW_BYTES + TX_BYTES : m == 7 ? 4 * 16384u : m == 8 ? TW_BYTES + 2 * TX_BYTES : 3 * 16384u;
    };
    auto dma_x = [&](int m) __attribute__((always_inline)) {
      const unsigned vo = xvo + blk(xg, m) * 256u;
      const unsigned m0 = ldsb + xreg(m) + wid * 2048u;
      dma(m0, vo, 0u);
      dma(m0 + 1024u, vo, 2u * COL_BYTES);
    };
    if (dmaw) { dma_x(0); dma_w(21 + wid); }
    else tail_wait(2u);
    if (wid == 7 && lane == 0) ctl[0] = ok ? 0u : 1u;
    asm volatile("s_waitcnt vmcnt(0) lgkmcnt(0)\n\ts_barrier" ::: "memory");
    STAMP(8 + 63);
    const rsrc_t ro = mk_rsrc(outp, OUT_BYTES);
    const unsigned ovo = (4 * (lane >> 4) * 4096u + col0 + wc * 64 + 4 * (lane & 15)) * 4u;
    const int mcnt = wr == 0 ? 10 : tall ? 9 : 8;
    typedef __attribute__((address_space(3))) h8 lh8;
    lchar* fb = lds + (lane >> 4) * B_COL + (wc * 64 + (lane & 15)) * 16;
    lchar* fb4 = fb + 16 * B_COL;
    asm volatile("" : "+v"(fb4));
    lchar* fa = lds + TW_BYTES + (lane >> 4) * 512 + wr * 256 + (lane & 15) * 16;
    lchar* fa3 = fa - TW_BYTES + 3 * 16384;
    asm volatile("" : "+v"(fa3));
    unsigned long long tt[10];
    h8 bw[7][4];
#pragma unroll
    for (int m = 0; m < 10; ++m) {
      if (dmaw) {
        if (m == 0) dma_x(1);
        else if (m < 5) { dma_x(2 * m); dma_x(2 * m + 1); }
      }
      if (m < mcnt) {
        const unsigned badv = ctl[0];
        h8 af[7];
        const int R = m == 0 ? 0 : m + 1 < 7 ? m + 1 : 7;
        auto ord = [&](int i) { return i < R ? 6 - i : i - R; };
        auto fetch = [&](int s) __attribute__((always_inline)) {
          af[s] = xreg(m) >= TW_BYTES ? *(lh8*)(fa + (xreg(m) - TW_BYTES) + s * 2048) : *(lh8*)(fa3 + (xreg(m) - 3 * 16384) + s * 2048);
          if (s < 7 - R) {
#pragma unroll
            for (int n = 0; n < 4; ++n) bw[s][n] = *(lh8*)((s < 4 ? fb : fb4) + (s & 3) * 4 * B_COL + n * 256);
          }
        };
        constexpr int WIN = TWIN;
#pragma unroll
        for (int i = 0; i < WIN; ++i) fetch(ord(i));
#pragma unroll
        for (int i = 0; i < 7; ++i) {
          if (i + WIN < 7) fetch(ord(i + WIN));
          SB;
#pragma unroll
          for (int n = 0; n < 4; ++n)
            acc[m][n] = __builtin_amdgcn_mfma_f32_16x16x32_f16(af[ord(i)], bw[ord(i)][n], acc[m][n], 0, 0, 0);
          SB;
        }
        const bool bad = rfl(badv) != 0;
        if (m < 4 && wid == 7) {
          tail_wait(12u << (2 * m));
          if (__builtin_expect(!ok, 0) && lane == 0) ctl[0] = 1u;
        }
        SB;
        const unsigned orow = ovo + blk(wr, m) * (16u * 16384u);
#pragma unroll
        for (int j = 0; j < 4; ++j) {
          f4 o = f4{acc[m][0][j], acc[m][1][j], acc[m][2][j], acc[m][3][j]};
          if (bad) o = f4{__builtin_nanf(""), __builtin_nanf(""), __builtin_nanf(""), __builtin_nanf("")};
          __builtin_amdgcn_raw_buffer_store_b128(__builtin_bit_cast(u4, o), ro, orow + j * 16384u, 0, OAUX);
        }
      }
      if (m < 4 || m == 5)
        asm volatile("s_waitcnt vmcnt(%0) lgkmcnt(0)\n\ts_barrier" :: "i"(m == 0 ? 4 : m == 1 ? 6 : m < 4 ? 12 : 8) : "memory");
      SB;
    }
  }
}

}

extern "C" void kernel_launch(int rank, int nranks, ncclComm_t comm,
                              hipStream_t stream,
                              void* const* d_in, const int* in_sizes, int n_in,
                              void* d_out, int out_size,
                              const int* in_dtypes, int out_dtype,
                              void* const* d_out_all,
                              void* const* d_ws_all, size_t ws_bytes) {
  if (nranks != NR || (unsigned)rank >= (unsigned)NR || ws_bytes < WS_NEED || n_in != 2 ||
      in_sizes[0] != 4096 * 512 || in_sizes[1] != 4096 * 512 || out_size != 4096 * 4096) {
    fprintf(stderr, "kernel_launch: unexpected problem shape\n");
    return;
  }
  Params p;
  p.x = (const float*)d_in[0];
  p.w = (const float*)d_in[1];
  p.out = (float*)d_out;
  p.me = (char*)d_ws_all[rank];
  for (int i = 0; i < 7; ++i) p.peer[i] = (char*)d_ws_all[(rank + 1 + i) % NR];
  hipLaunchKernelGGL(gemm_ar_kernel, dim3(NWG), dim3(512), 0, stream, p);
  HIP_CHECK(hipGetLastError());
}
```

```cpp
#include <hip/hip_runtime.h>
#include <rccl/rccl.h>
#include <cstdio>

#define HIP_CHECK(expr) do { hipError_t _e = (expr); if (_e != hipSuccess) { \
    fprintf(stderr, "HIP error %s at %s:%d\n", hipGetErrorString(_e), __FILE__, __LINE__); \
    return; } } while (0)

#define OAUX 16
#define SLEEP 2
#define JSW 96
#define TCMAP 3
#define TWIN 2
#define OD 6
#define OSFX " sc1"
#define NPUSH 3
#define EARLY 10
#define PACE 4
#define GDELAY0 30
#define GDELAY 30

namespace {

typedef _Float16 h8 __attribute__((ext_vector_type(8)));
typedef _Float16 h4 __attribute__((ext_vector_type(4)));
typedef float f4 __attribute__((ext_vector_type(4)));
typedef unsigned u4 __attribute__((ext_vector_type(4)));
typedef unsigned u2 __attribute__((ext_vector_type(2)));
typedef __attribute__((address_space(1))) unsigned gu32;
typedef __attribute__((address_space(3))) char lchar;
typedef __attribute__((address_space(3))) unsigned lu32;
typedef __amdgpu_buffer_rsrc_t rsrc_t;

constexpr int NR = 8;
constexpr int NCOMM = 32, NGEMM = 224, NWG = NCOMM + NGEMM;
constexpr unsigned SHARD_BYTES = 4096u * 512u * 4u;
constexpr unsigned OUT_BYTES = 4096u * 4096u * 4u;
constexpr unsigned COL_BYTES = 4096u * 16u;
constexpr unsigned OPER_BYTES = 512u * COL_BYTES;
constexpr unsigned BUF_BYTES = 2u * OPER_BYTES;
constexpr size_t CTRL_BYTES = 1u << 20;
constexpr size_t WS_NEED = CTRL_BYTES + 2ull * BUF_BYTES;
constexpr unsigned COL_A = 64, COL_B = 64 + 7 * NPUSH;
constexpr unsigned OFF_CALL = 0;
constexpr unsigned OFF_TMO = 0x8000;
constexpr unsigned OFF_QUEUE = 0x8100;
constexpr unsigned OFF_RDY = 0x8200;
constexpr unsigned OFF_OUTF = 0x28000;
constexpr unsigned OFF_RDY0 = 0x20000;
constexpr unsigned OFF_TOP = 0x8500;
constexpr unsigned OFF_GO = 0x8700;
constexpr unsigned OFF_PSH = 0x8400;
constexpr unsigned OFF_CNT1 = 0x8600;
constexpr unsigned OFF_OWN = 0x9000;
constexpr unsigned OFF_DUMMY = 0xC000;
constexpr unsigned OFF_FB = 0x10000;
constexpr unsigned FB_U = 384;
constexpr unsigned long long DEADLINE = 30000000ull;

constexpr int TM = 288, TN = 256;
constexpr unsigned A_COL = TM * 16, B_COL = TN * 16;
constexpr unsigned A_BYTES = 4 * A_COL, B_BYTES = 4 * B_COL, STAGE = A_BYTES + B_BYTES;
constexpr unsigned NMAIN = 121, COL_T = NMAIN * 4;
constexpr unsigned TW_BYTES = 28 * B_COL, TX_BYTES = 28 * 512;
constexpr unsigned LDS_CTL = TW_BYTES + 3 * TX_BYTES;
constexpr unsigned LDS_BYTES = LDS_CTL + 64;
static_assert(LDS_CTL >= 4 * STAGE && LDS_BYTES <= 160 * 1024, "LDS layout");
constexpr unsigned CV_Q = 1024 + 32, CV_WAVE = 4 * CV_Q;

constexpr int NKG = 64 - NPUSH, NT = NKG * 125;
constexpr unsigned SHB = 240;
constexpr int NP = 64;

template <int V> struct IC { static constexpr int value = V; };

struct Params {
  const float* x;
  const float* w;
  float* out;
  char* me;
  char* peer[7];
  char* pout[7];
  unsigned rank;
};

__device__ __forceinline__ rsrc_t mk_rsrc(const void* p, unsigned bytes) {
  return __builtin_amdgcn_make_buffer_rsrc((void*)p, (short)0, (int)bytes, 0x00020000);
}
__device__ __forceinline__ unsigned rfl(unsigned v) { return __builtin_amdgcn_readfirstlane(v); }

#define STAMPW(i, w) do {} while (0)
#define STAMPC(i) do {} while (0)
#define STAMP(i) STAMPW(i, 7)
#define TICK(x)
#define PUT(i, x)

template <class F>
__device__ __forceinline__ bool spin_until(char* me, unsigned call, unsigned lane, F cond) {
  gu32* tmo = (gu32*)(me + OFF_TMO);
  const unsigned long long t0 = __builtin_amdgcn_s_memrealtime();
  for (unsigned spin = 1;; ++spin) {
    if (cond()) return true;
    if ((spin & 63) == 0 && (__builtin_amdgcn_s_memrealtime() - t0 > DEADLINE ||
        rfl(__hip_atomic_load(tmo, __ATOMIC_RELAXED, __HIP_MEMORY_SCOPE_AGENT)) == call)) {
      if (lane == 0) __hip_atomic_store(tmo, call, __ATOMIC_RELAXED, __HIP_MEMORY_SCOPE_AGENT);
      return false;
    }
    __builtin_amdgcn_s_sleep(SLEEP);
  }
}

__global__ void __launch_bounds__(512, 2) gemm_ar_kernel(Params p) {
  __shared__ __attribute__((aligned(16))) char smem[LDS_BYTES];
  TICK(tk0);
  lchar* lds = (lchar*)smem;
  char* const me = p.me;
  float* const outp = p.out;
  const float* const xp = p.x;
  const float* const wp = p.w;
  const unsigned rank = p.rank & (NR - 1);
  const unsigned lane = threadIdx.x & 63;
  const unsigned wid = rfl(threadIdx.x >> 6);
  const unsigned prot = (blockIdx.x * 4 + wid) % 7;
  char* pptr[7];
  gu32* callp = (gu32*)(me + OFF_CALL + blockIdx.x * 128u);
  volatile lu32* ctl = (volatile lu32*)(lds + LDS_CTL);
  STAMP(0);
  const bool is_comm = blockIdx.x < NCOMM;
  const unsigned gi = blockIdx.x - NCOMM;
  const unsigned ctask = is_comm ? (wid >> 2) * (15u * 128u) + blockIdx.x * 4 + (wid & 3)
                       : wid >= 4 ? 256u + 768u + (wid - 4) * NGEMM + gi
                       : gi < 32 ? 128u + gi * 4 + wid : 256u + wid * 192u + (gi - 32);
  const bool urgent = ctask < 256;
  lchar* cvl = lds + wid * CV_WAVE;
  f4 v[8];
  auto cv_load = [&]() __attribute__((always_inline)) {
    const unsigned kb = ctask >> 7, unit = ctask & 63;
    const bool is_w = (ctask & 64) != 0;
    const rsrc_t src = mk_rsrc(is_w ? wp : xp, SHARD_BYTES);
    const unsigned voff = is_w ? (kb * 32u + (lane >> 4) * 8u) * 16384u + unit * 256u + (lane & 15) * 16u
                               : (unit * 64u + (lane >> 3)) * 2048u + kb * 128u + (lane & 7) * 16u;
#pragma unroll
    for (int i = 0; i < 8; ++i)
      v[i] = __builtin_bit_cast(f4, __builtin_amdgcn_raw_buffer_load_b128(src, voff, i * 16384u, 0));
  };
  if (!urgent) {
    if (threadIdx.x == 448) { ctl[1] = 0; ctl[2] = 0; ctl[3] = 0; }
    asm volatile("s_waitcnt lgkmcnt(0)\n\ts_barrier" ::: "memory");
    __builtin_amdgcn_s_sleep(GDELAY0);
    if (!is_comm)
      for (unsigned k = 3; k < (ctask >> 7); k += 2) __builtin_amdgcn_s_sleep(GDELAY);
  }
  cv_load();
  const unsigned sx = __builtin_amdgcn_raw_buffer_load_b32(mk_rsrc(xp, SHARD_BYTES), ((lane >> 4) * 1031u * 512u + (lane & 15)) * 4u, 0, 0) & 0x7fffffffu;
  const unsigned sw = __builtin_amdgcn_raw_buffer_load_b32(mk_rsrc(wp, SHARD_BYTES), ((lane >> 4) * (131u * 4096u + 1040u) + (lane & 15)) * 4u, 0, 0) & 0x7fffffffu;
  const unsigned callv = __hip_atomic_load(callp, __ATOMIC_RELAXED, __HIP_MEMORY_SCOPE_AGENT);
  if (urgent) asm volatile("s_barrier" ::: "memory");
#pragma unroll
  for (int i = 0; i < 7; ++i) pptr[i] = p.peer[prot + i >= 7 ? prot + i - 7 : prot + i];
  char* lpeer = me;
#pragma unroll
  for (int i = 0; i < 7; ++i) lpeer = lane == (unsigned)i ? p.peer[i] : lpeer;
  rsrc_t pr[7];
  unsigned pidx[7];
#pragma unroll
  for (int i = 0; i < 7; ++i) {
    pidx[i] = prot + i >= 7 ? prot + i - 7 : prot + i;
    pr[i] = mk_rsrc(pptr[i] + CTRL_BYTES, 2u * BUF_BYTES);
  }
  const rsrc_t own = mk_rsrc(me + CTRL_BYTES, 2u * BUF_BYTES);
  const unsigned call = rfl(callv) + 1;
  const unsigned par = (call & 1) * BUF_BYTES;
  auto arrive_rdy0 = [&]() {
    unsigned t = 0;
    if (lane == 0) t = __hip_atomic_fetch_add((lu32*)&ctl[2], 1u, __ATOMIC_RELAXED, __HIP_MEMORY_SCOPE_WORKGROUP);
    if (rfl(t) == 3 && lane < 8)
      __hip_atomic_store((gu32*)(lpeer + OFF_RDY0 + ((lane == 7 ? 7u : 6u - lane) * 32u + (blockIdx.x & 31)) * 128u + (is_comm ? 0u : 4u)),
                         call, __ATOMIC_RELAXED, __HIP_MEMORY_SCOPE_SYSTEM);
  };
  {
    constexpr unsigned LO = (127u - 8u) << 23, HI = (127u + 8u) << 23;
    if (__builtin_expect(!(__any(sx >= LO) && __any(sw >= LO) && __all(sx < HI) && __all(sw < HI)), 0)) {
      unsigned mx = sx, mw = sw;
#pragma unroll
      for (int o = 1; o < 64; o <<= 1) {
        mx = max(mx, (unsigned)__shfl_xor((int)mx, o));
        mw = max(mw, (unsigned)__shfl_xor((int)mw, o));
      }
      const int ex = (int)(rfl(mx) >> 23), ew = (int)(rfl(mw) >> 23);
      if (ex > 0 && ex < 255 && ew > 0 && ew < 255) {
        const int e = (ew - ex) >> 1;
        const int es = min(max((ctask & 64) ? -e : e, -126), 126);
        const float sc = __builtin_bit_cast(float, (unsigned)(127 + es) << 23);
#pragma unroll
        for (int i = 0; i < 8; ++i) v[i] *= sc;
      }
    }
  }
  {
    const unsigned kb = ctask >> 7;
    if (ctask & 64) {
#pragma unroll
      for (int u = 0; u < 4; ++u) {
        h8 g;
#pragma unroll
        for (int i = 0; i < 8; ++i) g[i] = (_Float16)v[i][u];
        *(__attribute__((address_space(3))) h8*)(cvl + (lane >> 4) * CV_Q + (16 * u + (lane & 15)) * 16) = g;
      }
    } else {
#pragma unroll
      for (int i = 0; i < 8; ++i) {
        h4 g;
#pragma unroll
        for (int u = 0; u < 4; ++u) g[u] = (_Float16)v[i][u];
        *(__attribute__((address_space(3))) h4*)(cvl + ((lane & 7) >> 1) * CV_Q + (8 * i + (lane >> 3)) * 16 + (lane & 1) * 8) = g;
      }
    }
    __builtin_amdgcn_fence(__ATOMIC_SEQ_CST, "wavefront");
    const unsigned cvo = par + (ctask & 64) * (OPER_BYTES / 64) + (ctask & 63) * 1024u + lane * 16u;
    u4 d[4];
#pragma unroll
    for (int q = 0; q < 4; ++q) d[q] = *(__attribute__((address_space(3))) u4*)(cvl + q * CV_Q + lane * 16);
    if (kb == 1) {
      TICK(tks);
#pragma unroll
      for (int q = 0; q < NPUSH; ++q) {
#pragma unroll
        for (int i = 0; i < 7; ++i) {
          __builtin_amdgcn_raw_buffer_store_b128(d[q], pr[i], cvo + (COL_A + 7u * q + 6u - pidx[i]) * COL_BYTES, 0, 17);
          asm volatile("s_waitcnt vmcnt(%0)" :: "i"(PACE) : "memory");
          if (q * 7 + i + 1 == NPUSH * 7 - EARLY) arrive_rdy0();
        }
      }
      TICK(tk4);
      PUT(120, tk0); PUT(123, tks); PUT(124, tk4);
    }
#pragma unroll
    for (int q = 0; q < 4; ++q) __builtin_amdgcn_raw_buffer_store_b128(d[q], own, cvo + (kb * 4 + q) * COL_BYTES, 0, 17);
    asm volatile("s_waitcnt vmcnt(0)" ::: "memory");
    if (kb == 1) {
      TICK(tk5);
      PUT(126, tk5);
      unsigned last = 0;
      if (lane == 0 && __hip_atomic_fetch_add((lu32*)&ctl[3], 1u, __ATOMIC_RELAXED, __HIP_MEMORY_SCOPE_WORKGROUP) == 3)
        last = (__hip_atomic_fetch_add((gu32*)(me + OFF_CNT1), 1u, __ATOMIC_RELAXED, __HIP_MEMORY_SCOPE_AGENT) & 31) == 31;
      if (rfl(last) && lane < 7)
        __hip_atomic_store((gu32*)(lpeer + OFF_PSH + (6u - lane) * 4u), call, __ATOMIC_RELAXED, __HIP_MEMORY_SCOPE_SYSTEM);
    }
    if (kb == 0) arrive_rdy0();
    unsigned last = 0;
    if (lane == 0 && __hip_atomic_fetch_add((lu32*)&ctl[1], 1u, __ATOMIC_RELAXED, __HIP_MEMORY_SCOPE_WORKGROUP) == 7u) {
      __hip_atomic_store(callp, call, __ATOMIC_RELAXED, __HIP_MEMORY_SCOPE_AGENT);
      if ((__hip_atomic_fetch_add((gu32*)(me + OFF_OWN + (blockIdx.x & 7) * 128u), 1u, __ATOMIC_RELAXED, __HIP_MEMORY_SCOPE_AGENT) & 31) == 31)
        last = (__hip_atomic_fetch_add((gu32*)(me + OFF_TOP), 1u, __ATOMIC_RELAXED, __HIP_MEMORY_SCOPE_AGENT) & 7) == 7;
    }
    if (rfl(last) && lane < 8)
      __hip_atomic_store((gu32*)(lpeer + (lane == 7 ? OFF_GO : OFF_RDY + (6u - lane) * 4u)), call, __ATOMIC_RELAXED, __HIP_MEMORY_SCOPE_SYSTEM);
  }
  const rsrc_t ctr = mk_rsrc(me, 0x30000u);

  if (blockIdx.x < NCOMM) {
    if (wid >= 4) return;
    bool ok = true;
    if (wid < 2) {
    const unsigned pw = wid * 32 + blockIdx.x;
    gu32* queue = (gu32*)(me + OFF_QUEUE);
    const unsigned qbase = (call - 1) * (unsigned)NT - 3u * NP;
    auto lookup = [&](unsigned n) -> unsigned {
      const unsigned kg = n / 125u, r = n - kg * 125u, i = n - (NT - 500u);
      unsigned u = (i - 272) >> 2;
      u += (u >= 6) + (u >= 12) + (u >= 18);
      const unsigned t = i < 16 ? (NKG - 4 + (i & 3)) * 128 + (i < 4 ? 60 : (i >> 2) * 7 - 1)
                       : i < 272 ? (NKG - 4 + ((i - 16) >> 6)) * 128 + 64 + ((i - 16) & 63) : (NKG - 4 + (i & 3)) * 128 + u;
      return n >= (unsigned)NT ? 0xffffu : n < NT - 500u ? kg * 128 + (r < 61 ? r : r + 3) : t;
    };
    ok = spin_until(me, call, lane, [&]() {
      unsigned r = 0;
#pragma unroll
      for (int k = 0; k < 4; ++k) {
        const u2 f = __builtin_amdgcn_raw_buffer_load_b64(ctr, OFF_RDY0 + k * 8192u + lane * 128u, 0, 17);
        r |= (f[0] - call) | (f[1] - call);
      }
      return __all((int)r >= 0); });
    STAMPW(3, 0);
    const unsigned lbase = par + lane * 16u;
    const unsigned lown = lane < 32 ? lbase + (SHB + 2 * rank) * 256u : 0x40000000u;
    auto unit_off = [&](unsigned task) {
      return (task & 127) == 60 ? lown : lbase + (task & 64) * (OPER_BYTES / 64) + (task & 63) * 1024u;
    };
    auto src_off = [&](unsigned task) {
      return unit_off(task) + ((task >> 7) + (task >= 512 ? NPUSH : 0)) * COL_BYTES + (task == 0xffff ? 0x80000000u : 0u);
    };
    const unsigned roff = OFF_RDY + min(lane, 6u) * 4u;
    const unsigned one = 1;
    unsigned prev = 0xffff, cur = ok ? lookup(pw) : 0xffffu;
    bool ready = false;
    u4 v[7];
    {
      const unsigned o = src_off(cur);
#pragma unroll
      for (int i = 0; i < 7; ++i)
        asm volatile("buffer_load_dwordx4 %0, %1, %2, 0 offen sc0 sc1" : "=v"(v[i]) : "v"(o), "s"(pr[i]) : "memory");
    }
    asm volatile("s_waitcnt vmcnt(0)" ::: "memory");
    unsigned it = 0;
    auto round = [&](unsigned& tk, unsigned& rv) {
      const unsigned od = unit_off(cur) + (COL_B + (cur >> 7) * 7u) * COL_BYTES;
      asm volatile("s_waitcnt vmcnt(15)" : "+v"(tk), "+v"(rv) :: "memory");
      unsigned nxt = lookup(rfl(tk) - qbase);
      if (!ready) {
        ready = __all((int)(rv - call) >= 0);
        if (!ready && nxt != 0xffff && nxt >= 4 * 128) {
          ok = spin_until(me, call, lane, [&]() {
            return __all((int)(__builtin_amdgcn_raw_buffer_load_b32(ctr, roff, 0, 17) - call) >= 0); });
          ready = true;
          if (!ok) nxt = 0xffff;
        }
      }
      const unsigned os = src_off(nxt);
#pragma unroll
      for (int i = 0; i < 7; ++i) {
        if (i) asm volatile("s_waitcnt vmcnt(15)" ::: "memory");
        asm volatile("buffer_store_dwordx4 %0, %1, %2, %3 offen sc1\n\tbuffer_load_dwordx4 %0, %4, %5, 0 offen sc0 sc1"
                     : "+v"(v[i]) : "v"(od), "s"(own), "s"(pidx[i] * COL_BYTES), "v"(os), "s"(pr[i]) : "memory");
      }
      const unsigned fo = prev == 0xffff ? OFF_DUMMY + pw * 64u : OFF_FB + (prev & 127) * FB_U + (prev >> 7) * 4u;
      if (lane == 0) {
        asm volatile("buffer_store_dword %0, %1, %2, 0 offen sc1" :: "v"(call), "v"(fo), "s"(ctr) : "memory");
        asm volatile("global_atomic_add %0, %1, %2, off sc0" : "+v"(tk) : "v"(queue), "v"(one) : "memory");
      }
      asm volatile("buffer_load_dword %0, %1, %2, 0 offen sc0 sc1" : "+v"(rv) : "v"(roff), "s"(ctr) : "memory");
      prev = cur; cur = nxt;
    };
    unsigned tka = qbase + NP + pw, tkb = qbase + 2 * NP + pw, rva = call - 1, rvb = call - 1;
    while (cur != 0xffff) {
      round(tka, rva);
      if (cur == 0xffff) break;
      round(tkb, rvb);
    }
    asm volatile("s_waitcnt vmcnt(4)" ::: "memory");
    if (lane == 0 && prev != 0xffff)
      __hip_atomic_store((gu32*)(me + OFF_FB + (prev & 127) * FB_U + (prev >> 7) * 4u), call, __ATOMIC_RELAXED, __HIP_MEMORY_SCOPE_AGENT);
    asm volatile("s_waitcnt vmcnt(0)" :: "v"(tka), "v"(tkb), "v"(rva), "v"(rvb), "v"(v[0]), "v"(v[1]), "v"(v[2]), "v"(v[3]),
                 "v"(v[4]), "v"(v[5]), "v"(v[6]) : "memory");
    STAMPW(1, 0);
    if (lane == 0) ctl[3] = 1;
    } else {
      while (rfl(ctl[3]) == 0) __builtin_amdgcn_s_sleep(16);
    }
    if (blockIdx.x < 28) {
      const unsigned item = wid * 28 + blockIdx.x, otc = item / 7, pi = item - otc * 7;
      const unsigned so = (SHB + 2 * ((rank + 1 + pi) & (NR - 1))) * (16u * 16384u);
      const rsrc_t po = mk_rsrc(p.pout[pi] + so, 32u * 16384u);
      const rsrc_t ro = mk_rsrc((char*)outp + so, 32u * 16384u);
      if (ok)
        ok = spin_until(me, call, lane, [&]() {
          return (int)(rfl(__builtin_amdgcn_raw_buffer_load_b32(ctr, OFF_OUTF + (pi * 16u + otc) * 128u, 0, 17)) - call) >= 0; });
      STAMPW(5, 0);
      const unsigned o = otc * 1024u + lane * 16u;
      if (ok) {
        u4 w[OD];
#pragma unroll
        for (int i = 0; i < OD; ++i)
          asm volatile("buffer_load_dwordx4 %0, %1, %2, %3 offen sc0 sc1" : "=v"(w[i]) : "v"(o), "s"(po), "s"(i * 16384u) : "memory");
#pragma unroll
        for (int i = 0; i < OD; ++i)
          asm volatile("s_waitcnt vmcnt(%6)\n\tbuffer_store_dwordx4 %0, %1, %2, %3 offen" OSFX "\n\tbuffer_load_dwordx4 %0, %1, %4, %5 offen sc0 sc1"
                       : "+v"(w[i]) : "v"(o), "s"(ro), "s"(i * 16384u), "s"(po), "s"((i + OD) * 16384u), "i"(OD - 1 + i) : "memory");
        for (unsigned r = OD; r < 32; r += OD) {
#pragma unroll
          for (int i = 0; i < OD; ++i)
            asm volatile("s_waitcnt vmcnt(%6)\n\tbuffer_store_dwordx4 %0, %1, %2, %3 offen" OSFX "\n\tbuffer_load_dwordx4 %0, %1, %4, %5 offen sc0 sc1"
                         : "+v"(w[i]) : "v"(o), "s"(ro), "s"((r + i) * 16384u), "s"(po), "s"((r + i + OD) * 16384u), "i"(2 * OD - 2) : "memory");
        }
        asm volatile("s_waitcnt vmcnt(0)" :: "v"(w[0]), "v"(w[1]), "v"(w[2]), "v"(w[3]), "v"(w[OD - 4]), "v"(w[OD - 3]), "v"(w[OD - 2]), "v"(w[OD - 1]) : "memory");
      } else {
        const u4 nan4 = {0x7fc00000u, 0x7fc00000u, 0x7fc00000u, 0x7fc00000u};
        for (int r = 0; r < 32; ++r) __builtin_amdgcn_raw_buffer_store_b128(nan4, ro, o + r * 16384u, 0, 16);
      }
      STAMPW(6, 0);
    }
    return;
  }

  const unsigned xcd = gi & 7, ti7 = gi >> 5, ti = (xcd & 1) * 7 + ti7;
  const unsigned tp = xcd >> 1, tq = (gi >> 3) & 3;
  const unsigned tc = TCMAP == 0 ? tp * 4 + tq : TCMAP == 1 ? tq * 4 + tp : TCMAP == 2 ? tp * 2 + (tq & 1) + (tq >> 1) * 8 : tq * 4 + ((tp + tq) & 3), col0 = tc * TN;
  const bool tall = ti7 >= 5;
  const unsigned p8 = tall ? 224 + 2 * ((xcd & 1) * 2 + ti7 - 5) : 232 + (xcd & 1) * 5 + ti7;
  auto blk = [&](unsigned g, unsigned m) -> unsigned {
    const unsigned l = m < 8 ? 28 * m + 2 * ti + g : p8 + (tall ? g : 0u);
    return l < 2 ? SHB + 2 * rank + l : l - 2;
  };
  STAMP(1);

  const rsrc_t stg = mk_rsrc(me + CTRL_BYTES + par, BUF_BYTES);
  constexpr int ND0 = 9;
  const unsigned pc0 = min(wid * ND0, 34u - ND0);
  unsigned dvo[ND0];
#pragma unroll
  for (int i = 0; i < ND0; ++i) {
    const unsigned pc = pc0 + i;
    if (pc < 18) {
      const unsigned e = pc * 64 + lane, q = e / TM, r = e - q * TM, g = r >= 144, m = (r - 144 * g) >> 4;
      dvo[i] = q * COL_BYTES + (blk(g, m) * 16u + (r & 15)) * 16u;
    } else {
      const unsigned e = (pc - 18) * 64 + lane, q = e >> 8, r = e & 255;
      dvo[i] = OPER_BYTES + q * COL_BYTES + (col0 + r) * 16u;
    }
  }
  const unsigned ldsb = (unsigned)(size_t)lds;
  auto dma = [&](unsigned m0, unsigned vo, unsigned so) __attribute__((always_inline)) {
    asm volatile("s_mov_b32 m0, %0\n\ts_nop 0\n\tbuffer_load_dwordx4 %1, %2, %3 offen sc1 lds"
                 :: "s"(m0), "v"(vo), "s"(stg), "s"(so) : "memory");
  };

  unsigned poff;
  {
    const unsigned wk = min(lane / 14, 3u), wh = lane % 14;
    const unsigned u = wh < 10 ? min(blk(0, wh) >> 2, 60u) : 64 + tc * 4 + (wh - 10);
    poff = OFF_FB + u * FB_U + wk * 4u;
  }
  unsigned kg0 = 0;
  unsigned avail = COL_A;
  bool ok = true;
  auto poll_issue = [&]() -> unsigned { return __builtin_amdgcn_raw_buffer_load_b32(ctr, poff, kg0 * 4u, 16); };
  auto poll_eval = [&](unsigned v) {
    const unsigned long long m = __ballot((int)(v - call) >= 0) & 0x00ffffffffffffffull;
    kg0 = min(kg0 + (unsigned)__builtin_ctzll(~m) / 14u, (unsigned)NKG - 4);
    avail = COL_B + kg0 * 7;
  };

  const unsigned wr = wid >> 2, wc = wid & 3;
  const unsigned a_off = (lane >> 4) * A_COL + (wr * 144 + (lane & 15)) * 16;
  const unsigned b_off = A_BYTES + (lane >> 4) * B_COL + (wc * 64 + (lane & 15)) * 16;
  f4 acc[9][4];
#pragma unroll
  for (int m = 0; m < 9; ++m)
#pragma unroll
    for (int n = 0; n < 4; ++n) acc[m][n] = f4{0.f, 0.f, 0.f, 0.f};

  unsigned pv = 0, pend = 0;
  if (wid == 7)
    ok = spin_until(me, call, lane, [&]() { return (int)(rfl(__builtin_amdgcn_raw_buffer_load_b32(ctr, OFF_GO, 0, 17)) - call) >= 0; });
  asm volatile("s_barrier" ::: "memory");
  STAMP(2);
  STAMPC(5);
  unsigned jev = COL_A / 4 - 4;
  auto poll_step = [&](unsigned j) __attribute__((always_inline)) {
    const unsigned need = (j + (j < JSW ? 4 : 3)) * 4;
    if (!ok) {
    } else if (avail == COL_A) {
      if (need > COL_A) {
        ok = spin_until(me, call, lane, [&]() {
          return __all((int)(__builtin_amdgcn_raw_buffer_load_b32(ctr, OFF_PSH + min(lane, 6u) * 4u, 0, 17) - call) >= 0); });
        avail = COL_B;
      }
    } else {
      if (pend) poll_eval(pv);
      if (avail < need) ok = spin_until(me, call, lane, [&]() { poll_eval(poll_issue()); return avail >= need; });
    }
    asm volatile("s_waitcnt lgkmcnt(0)" ::: "memory");
    const unsigned jblk = avail / 4 - (avail / 4 < JSW + 3 ? 3 : 2);
    pend = 0;
    if (!ok || avail >= COL_T) {
      jev = 1000;
    } else if (avail != COL_A && jblk <= j + 4) {
      pv = poll_issue(); pend = 1;
      jev = min(jblk, j + 2);
    } else {
      jev = avail == COL_A ? jblk : jblk - 4;
    }
  };
  const unsigned la = ldsb + a_off, lb = ldsb + b_off;
  h8 a[9], b[4];
  auto kloop = [&](auto mu_c, auto role_c) __attribute__((always_inline)) {
    constexpr int MU = decltype(mu_c)::value;
    constexpr int ROLE = decltype(role_c)::value;
#define RD(dst, base, off) asm volatile("ds_read_b128 %0, %1 offset:%2" : "+v"(dst) : "v"(base), "i"(off) : "memory")
#define SB __builtin_amdgcn_sched_barrier(0)
    const unsigned lds0 = ldsb + pc0 * 1024u;
    if (ROLE == 0) {
#pragma unroll
      for (int st = 0; st < 3; ++st)
#pragma unroll
        for (int i = 0; i < ND0; ++i) dma(lds0 + st * STAGE + i * 1024u, dvo[i], st * (4u * COL_BYTES));
      asm volatile("s_waitcnt vmcnt(%0)" :: "i"(2 * ND0) : "memory");
    }
    asm volatile("s_barrier" ::: "memory");
    RD(b[0], lb, 0); RD(b[1], lb, 256);
#pragma unroll
    for (int m = 0; m < MU; ++m) RD(a[m], la, m * 256);
    RD(b[2], lb, 512); RD(b[3], lb, 768);
    if (ROLE != 0) __builtin_amdgcn_s_setprio(1);
    for (unsigned j = 0; j < NMAIN; ++j) {
      if (ROLE == 2 && __builtin_expect(j >= jev, 0)) poll_step(j);
      if (ROLE != 0) asm volatile("s_barrier" ::: "memory");
      const unsigned sd = j < JSW ? j + 3 : j + 2;
      const unsigned m0 = lds0 + (sd & 3) * STAGE;
      const unsigned so = sd * (4u * COL_BYTES);
      const unsigned nb = ((j + 1) & 3) * STAGE;
      const unsigned na = la + nb, nbb = lb + nb;
      SB;
#pragma unroll
      for (int m = 0; m < MU; ++m) {
        if (m == 0) asm volatile("s_waitcnt lgkmcnt(%3)" : "+v"(a[0]), "+v"(b[0]), "+v"(b[1]) : "i"(MU + 1) : "memory");
        else asm volatile("s_waitcnt lgkmcnt(%1)" : "+v"(a[m]) : "i"(MU + 1 - m) : "memory");
        acc[m][0] = __builtin_amdgcn_mfma_f32_16x16x32_f16(a[m], b[0], acc[m][0], 0, 0, 0);
        acc[m][1] = __builtin_amdgcn_mfma_f32_16x16x32_f16(a[m], b[1], acc[m][1], 0, 0, 0);
        SB;
      }
      if (ROLE == 0) {
        if (j > JSW) asm volatile("s_waitcnt vmcnt(0)" ::: "memory");
        asm volatile("s_waitcnt vmcnt(%0)\n\ts_barrier" :: "i"(ND0) : "memory");
      }
      RD(b[0], nbb, 0); RD(b[1], nbb, 256);
      if (ROLE == 0 && j != JSW && sd < NMAIN) {
#pragma unroll
        for (int i = 0; i < ND0; ++i) dma(m0 + i * 1024u, dvo[i], so);
      }
      asm volatile("s_waitcnt lgkmcnt(2)" : "+v"(b[2]), "+v"(b[3]) :: "memory");
      SB;
#pragma unroll
      for (int m = 0; m < MU; ++m) {
        acc[m][2] = __builtin_amdgcn_mfma_f32_16x16x32_f16(a[m], b[2], acc[m][2], 0, 0, 0);
        acc[m][3] = __builtin_amdgcn_mfma_f32_16x16x32_f16(a[m], b[3], acc[m][3], 0, 0, 0);
        SB;
        RD(a[m], na, m * 256);
        SB;
      }
      RD(b[2], nbb, 512); RD(b[3], nbb, 768);
      SB;
    }
    asm volatile("s_waitcnt lgkmcnt(0)" :: "v"(a[0]), "v"(a[1]), "v"(a[2]), "v"(a[3]), "v"(a[4]), "v"(a[5]),
                 "v"(a[6]), "v"(a[7]), "v"(a[MU - 2]), "v"(a[MU - 1]), "v"(b[0]), "v"(b[1]), "v"(b[2]), "v"(b[3]) : "memory");
    if (ROLE != 0) __builtin_amdgcn_s_setprio(0);
  };
  if (wid < 4) kloop(IC<9>{}, IC<0>{});
  else if (!tall && wid < 7) kloop(IC<8>{}, IC<1>{});
  else if (!tall) kloop(IC<8>{}, IC<2>{});
  else if (wid < 7) kloop(IC<9>{}, IC<1>{});
  else kloop(IC<9>{}, IC<2>{});
  STAMP(3);
  STAMPC(6);

  {
    unsigned have = 0;
    auto tail_eval = [&](unsigned v) {
      const unsigned long long m = __ballot((int)(v - call) >= 0);
      const unsigned h3 = (unsigned)(m & (m >> 14) & (m >> 28));
      have = (h3 & (unsigned)(m >> 42) & 0x3fffu) | ((h3 & 0x3c00u) << 4);
    };
    auto tail_wait = [&](unsigned mask) __attribute__((always_inline)) {
      if (__builtin_expect(ok && (have & mask) != mask, 0))
        ok = spin_until(me, call, lane, [&]() {
          tail_eval(__builtin_amdgcn_raw_buffer_load_b32(ctr, poff, (NKG - 4) * 4u, 16)); return (have & mask) == mask; });
    };
    const bool dmaw = wid < 7;
    const unsigned wvo = OPER_BYTES + (col0 + lane) * 16u;
    auto dma_w = [&](unsigned c) __attribute__((always_inline)) {
#pragma unroll
      for (int i = 0; i < 4; ++i) dma(ldsb + c * B_COL + i * 1024u, wvo, (COL_T + c) * COL_BYTES + i * 1024u);
    };
    if (wid == 7) tail_wait(0x3c000u);
    asm volatile("s_barrier" ::: "memory");
    STAMP(8 + 60);
    if (dmaw) { dma_w(wid); dma_w(7 + wid); dma_w(14 + wid); }
    else tail_wait(0x3c01u);
    asm volatile("s_barrier" ::: "memory");
    STAMP(8 + 61);
    const unsigned xg = (lane >> 4) & 1;
    const unsigned xvo = (COL_T + wid * 4 + (lane >> 5)) * COL_BYTES + (lane & 15) * 16u;
    auto xreg = [](int m) -> unsigned {
      return m < 3 ? TW_BYTES + m * TX_BYTES : m == 3 ? TW_BYTES : m == 4 ? 6 * 16384u : m == 5 ? 5 * 16384u
           : m == 6 ? TW_BYTES + TX_BYTES : m == 7 ? 4 * 16384u : m == 8 ? TW_BYTES + 2 * TX_BYTES : 3 * 16384u;
    };
    auto dma_x = [&](int m) __attribute__((always_inline)) {
      const unsigned vo = xvo + blk(xg, m) * 256u;
      const unsigned m0 = ldsb + xreg(m) + wid * 2048u;
      dma(m0, vo, 0u);
      dma(m0 + 1024u, vo, 2u * COL_BYTES);
    };
    if (dmaw) { dma_x(0); dma_w(21 + wid); }
    else tail_wait(2u);
    if (wid == 7 && lane == 0) ctl[0] = ok ? 0u : 1u;
    asm volatile("s_waitcnt vmcnt(0) lgkmcnt(0)\n\ts_barrier" ::: "memory");
    STAMP(8 + 63);
    const rsrc_t ro = mk_rsrc(outp, OUT_BYTES);
    const unsigned ovo = (4 * (lane >> 4) * 4096u + col0 + wc * 64 + 4 * (lane & 15)) * 4u;
    const int mcnt = wr == 0 || tall ? 9 : 8;
    typedef __attribute__((address_space(3))) h8 lh8;
    lchar* fb = lds + (lane >> 4) * B_COL + (wc * 64 + (lane & 15)) * 16;
    lchar* fb4 = fb + 16 * B_COL;
    asm volatile("" : "+v"(fb4));
    lchar* fa = lds + TW_BYTES + (lane >> 4) * 512 + wr * 256 + (lane & 15) * 16;
    lchar* fa3 = fa - TW_BYTES + 3 * 16384;
    asm volatile("" : "+v"(fa3));
    unsigned long long tt[10];
    h8 bw[7][4];
#pragma unroll
    for (int m = 0; m < 9; ++m) {
      if (dmaw) {
        if (m == 0) dma_x(1);
        else if (m < 4) { dma_x(2 * m); dma_x(2 * m + 1); }
        else if (m == 4) dma_x(8);
      }
      if (m < mcnt) {
        const unsigned badv = ctl[0];
        h8 af[7];
        const int R = m == 0 ? 0 : m + 2 < 7 ? m + 2 : 7;
        auto ord = [&](int i) { return i < R ? 6 - i : i - R; };
        auto fetch = [&](int s) __attribute__((always_inline)) {
          af[s] = xreg(m) >= TW_BYTES ? *(lh8*)(fa + (xreg(m) - TW_BYTES) + s * 2048) : *(lh8*)(fa3 + (xreg(m) - 3 * 16384) + s * 2048);
          if (s < 7 - R) {
#pragma unroll
            for (int n = 0; n < 4; ++n) bw[s][n] = *(lh8*)((s < 4 ? fb : fb4) + (s & 3) * 4 * B_COL + n * 256);
          }
        };
        constexpr int WIN = TWIN;
#pragma unroll
        for (int i = 0; i < WIN; ++i) fetch(ord(i));
#pragma unroll
        for (int i = 0; i < 7; ++i) {
          if (i + WIN < 7) fetch(ord(i + WIN));
          SB;
#pragma unroll
          for (int n = 0; n < 4; ++n)
            acc[m][n] = __builtin_amdgcn_mfma_f32_16x16x32_f16(af[ord(i)], bw[ord(i)][n], acc[m][n], 0, 0, 0);
          SB;
        }
        const bool bad = rfl(badv) != 0;
        if (m < 4 && wid == 7) {
          tail_wait(12u << (2 * m));
          if (__builtin_expect(!ok, 0) && lane == 0) ctl[0] = 1u;
        }
        SB;
        const unsigned orow = ovo + blk(wr, m) * (16u * 16384u);
#pragma unroll
        for (int j = 0; j < 4; ++j) {
          f4 o = f4{acc[m][0][j], acc[m][1][j], acc[m][2][j], acc[m][3][j]};
          if (bad) o = f4{__builtin_nanf(""), __builtin_nanf(""), __builtin_nanf(""), __builtin_nanf("")};
          __builtin_amdgcn_raw_buffer_store_b128(__builtin_bit_cast(u4, o), ro, orow + j * 16384u, 0, 17);
        }
      }
      if (m == 0 && ti == 0) asm volatile("s_waitcnt vmcnt(0)" ::: "memory");
      if (m < 4 || m == 5)
        asm volatile("s_waitcnt vmcnt(%0) lgkmcnt(0)\n\ts_barrier" :: "i"(m == 0 ? 4 : m == 1 ? 6 : m < 4 ? 12 : 8) : "memory");
      if (m == 0 && ti == 0 && wid == 7 && lane < 7) {
        char* fp = p.peer[0];
#pragma unroll
        for (int i = 1; i < 7; ++i) fp = lane == (unsigned)i ? p.peer[i] : fp;
        __hip_atomic_store((gu32*)(fp + OFF_OUTF + ((6u - lane) * 16u + tc) * 128u), call, __ATOMIC_RELAXED, __HIP_MEMORY_SCOPE_SYSTEM);
      }
      SB;
    }
  }
}

}

extern "C" void kernel_launch(int rank, int nranks, ncclComm_t comm,
                              hipStream_t stream,
                              void* const* d_in, const int* in_sizes, int n_in,
                              void* d_out, int out_size,
                              const int* in_dtypes, int out_dtype,
                              void* const* d_out_all,
                              void* const* d_ws_all, size_t ws_bytes) {
  if (nranks != NR || (unsigned)rank >= (unsigned)NR || ws_bytes < WS_NEED || n_in != 2 ||
      in_sizes[0] != 4096 * 512 || in_sizes[1] != 4096 * 512 || out_size != 4096 * 4096) {
    fprintf(stderr, "kernel_launch: unexpected problem shape\n");
    return;
  }
  Params p;
  p.x = (const float*)d_in[0];
  p.w = (const float*)d_in[1];
  p.out = (float*)d_out;
  p.me = (char*)d_ws_all[rank];
  for (int i = 0; i < 7; ++i) {
    p.peer[i] = (char*)d_ws_all[(rank + 1 + i) % NR];
    p.pout[i] = (char*)d_out_all[(rank + 1 + i) % NR];
  }
  p.rank = (unsigned)rank;
  hipLaunchKernelGGL(gemm_ar_kernel, dim3(NWG), dim3(512), 0, stream, p);
  HIP_CHECK(hipGetLastError());
}
```

```cpp
#include <hip/hip_runtime.h>
#include <rccl/rccl.h>
#include <cstdio>

#define HIP_CHECK(expr) do { hipError_t _e = (expr); if (_e != hipSuccess) { \
    fprintf(stderr, "HIP error %s at %s:%d\n", hipGetErrorString(_e), __FILE__, __LINE__); \
    return; } } while (0)

namespace {

typedef _Float16 h8 __attribute__((ext_vector_type(8)));
typedef _Float16 h4 __attribute__((ext_vector_type(4)));
typedef float f4 __attribute__((ext_vector_type(4)));
typedef unsigned u4 __attribute__((ext_vector_type(4)));
typedef unsigned u2 __attribute__((ext_vector_type(2)));
typedef __attribute__((address_space(1))) unsigned gu32;
typedef __attribute__((address_space(3))) char lchar;
typedef __attribute__((address_space(3))) unsigned lu32;
typedef __amdgpu_buffer_rsrc_t rsrc_t;

constexpr int NPUSH = 3;
constexpr int PACE = 4;
constexpr int EARLY = 10;
constexpr int GDELAY = 30;
constexpr unsigned JSW = 96;
constexpr int TWIN = 2;
constexpr int OD = 6;

constexpr int NR = 8;
constexpr int NCOMM = 32, NGEMM = 224, NWG = NCOMM + NGEMM;
constexpr unsigned SHARD_BYTES = 4096u * 512u * 4u;
constexpr unsigned OUT_BYTES = 4096u * 4096u * 4u;
constexpr unsigned OPER_BYTES = 4096u * 16u;
constexpr unsigned PAR_BYTES = 2u * OPER_BYTES;
constexpr unsigned COL_BYTES = 2u * PAR_BYTES;
constexpr unsigned STG_BYTES = 512u * COL_BYTES;
constexpr size_t CTRL_BYTES = 1u << 20;
constexpr size_t WS_NEED = CTRL_BYTES + STG_BYTES;
constexpr unsigned COL_A = 64, COL_B = 64 + 7 * NPUSH;
constexpr unsigned OFF_CALL = 0;
constexpr unsigned OFF_TMO = 0x8000;
constexpr unsigned OFF_QUEUE = 0x8100;
constexpr unsigned OFF_RDY = 0x8200;
constexpr unsigned OFF_OUTF = 0x28000;
constexpr unsigned OFF_RDY0 = 0x20000;
constexpr unsigned OFF_TOP = 0x8500;
constexpr unsigned OFF_GO = 0x8700;
constexpr unsigned OFF_PSH = 0x8400;
constexpr unsigned OFF_CNT1 = 0x8600;
constexpr unsigned OFF_OWN = 0x9000;
constexpr unsigned OFF_DUMMY = 0xC000;
constexpr unsigned OFF_FB = 0x10000;
constexpr unsigned FB_U = 384;
constexpr unsigned long long DEADLINE = 100000000ull;

constexpr int TM = 288, TN = 256;
constexpr unsigned A_COL = TM * 16, B_COL = TN * 16;
constexpr unsigned A_BYTES = 4 * A_COL, B_BYTES = 4 * B_COL, STAGE = A_BYTES + B_BYTES;
constexpr unsigned NMAIN = 121, COL_T = NMAIN * 4;
constexpr unsigned TW_BYTES = 28 * B_COL, TX_BYTES = 28 * 512;
constexpr unsigned LDS_CTL = TW_BYTES + 3 * TX_BYTES;
constexpr unsigned LDS_BYTES = LDS_CTL + 64;
static_assert(LDS_CTL >= 4 * STAGE && LDS_BYTES <= 160 * 1024, "LDS layout");
constexpr unsigned CV_Q = 1024 + 32, CV_WAVE = 4 * CV_Q;

constexpr int NKG = 64 - NPUSH, NT = NKG * 125;
constexpr unsigned SHB = 240;
constexpr int NP = 64;

template <int V> struct IC { static constexpr int value = V; };

struct Params {
  const float* x;
  const float* w;
  float* out;
  char* me;
  char* peer[14];
  char* pout[7];
  unsigned rank;
  const char* warm[64];
};

__device__ __forceinline__ rsrc_t mk_rsrc(const void* p, unsigned bytes) {
  return __builtin_amdgcn_make_buffer_rsrc((void*)p, (short)0, (int)bytes, 0x00020000);
}
__device__ __forceinline__ unsigned rfl(unsigned v) { return __builtin_amdgcn_readfirstlane(v); }

template <class F>
__device__ __forceinline__ bool spin_until(char* me, unsigned call, unsigned lane, F cond) {
  gu32* tmo = (gu32*)(me + OFF_TMO);
  const unsigned long long t0 = __builtin_amdgcn_s_memrealtime();
  for (unsigned spin = 1;; ++spin) {
    if (cond()) return true;
    if ((spin & 63) == 0 && (__builtin_amdgcn_s_memrealtime() - t0 > DEADLINE ||
        rfl(__hip_atomic_load(tmo, __ATOMIC_RELAXED, __HIP_MEMORY_SCOPE_AGENT)) == call)) {
      if (lane == 0) __hip_atomic_store(tmo, call, __ATOMIC_RELAXED, __HIP_MEMORY_SCOPE_AGENT);
      return false;
    }
    __builtin_amdgcn_s_sleep(2);
    asm volatile("" ::: "memory");
  }
}

__global__ void __launch_bounds__(512, 2) gemm_ar_kernel(Params p) {
  __shared__ __attribute__((aligned(16))) char smem[LDS_BYTES];
  unsigned ka0, ka1, ka2;
  asm volatile("s_load_dword %0, %3, 0x0\n\ts_load_dword %1, %3, 0x40\n\ts_load_dword %2, %3, 0x80"
               : "=&s"(ka0), "=&s"(ka1), "=&s"(ka2) : "s"(__builtin_amdgcn_kernarg_segment_ptr()));
  unsigned long long pc_entry;
  asm volatile("s_getpc_b64 %0\n.Lgemm_ar_code_begin:" : "=s"(pc_entry));
  lchar* lds = (lchar*)smem;
  char* const me = p.me;
  float* const outp = p.out;
  const float* const xp = p.x;
  const float* const wp = p.w;
  const unsigned rank = p.rank & (NR - 1);
  const unsigned lane = threadIdx.x & 63;
  const unsigned wid = rfl(threadIdx.x >> 6);
  const unsigned prot = (blockIdx.x * 4 + wid) % 7;
  char* pptr[7];
  gu32* callp = (gu32*)(me + OFF_CALL + blockIdx.x * 128u);
  volatile lu32* ctl = (volatile lu32*)(lds + LDS_CTL);
  if (blockIdx.x >= NWG - 8 && wid == 7) {
    unsigned t;
    asm volatile("global_load_dword %0, %1, off sc0 sc1\n\ts_waitcnt vmcnt(0)" : "=&v"(t) : "v"(p.warm[lane]) : "memory");
  }
  const bool is_comm = blockIdx.x < NCOMM;
  const unsigned gi = blockIdx.x - NCOMM;
  const bool hi = wid >= 4, first = blockIdx.x < NCOMM + 32;
  const unsigned tbase = is_comm ? (hi ? 15u * 128u - 4u : 0u) : hi ? 1024u - 4u * NGEMM - NCOMM : first ? 0u : 256u - 64u;
  const unsigned tmul = is_comm || (first && !hi) ? 1u : hi ? NGEMM : 192u;
  const unsigned ctask = tbase + wid * tmul + (tmul == 1u ? blockIdx.x * 4 : blockIdx.x);
  const bool urgent = ctask < 256;
  lchar* cvl = lds + wid * CV_WAVE;
  f4 v[8];
  auto cv_load = [&]() __attribute__((always_inline)) {
    const unsigned kb = ctask >> 7, unit = ctask & 63;
    const bool is_w = (ctask & 64) != 0;
    const rsrc_t src = mk_rsrc(is_w ? wp : xp, SHARD_BYTES);
    const unsigned sh = is_w ? 4u : 3u;
    const unsigned voff = (is_w ? kb * (32u * 16384u) + unit * 256u : unit * (64u * 2048u) + kb * 128u)
                        + ((lane >> sh) << (is_w ? 17u : 11u)) + (lane & ((1u << sh) - 1u)) * 16u;
#pragma unroll
    for (int i = 0; i < 8; ++i)
      v[i] = __builtin_bit_cast(f4, __builtin_amdgcn_raw_buffer_load_b128(src, voff, i * 16384u, 0));
  };
  if (__builtin_expect(!urgent, 0)) {
    if (threadIdx.x == 448) { ctl[1] = 0; ctl[2] = 0; ctl[3] = 0; }
    asm volatile("s_waitcnt lgkmcnt(0)\n\ts_barrier" ::: "memory");
    __builtin_amdgcn_s_sleep(GDELAY);
    if (!is_comm)
      for (unsigned k = 3; k < (ctask >> 7); k += 2) __builtin_amdgcn_s_sleep(GDELAY);
  }
  cv_load();
  unsigned sx = __builtin_amdgcn_raw_buffer_load_b32(mk_rsrc(xp, SHARD_BYTES), ((lane >> 4) * 1031u * 512u + (lane & 15)) * 4u, 0, 0);
  unsigned sw = __builtin_amdgcn_raw_buffer_load_b32(mk_rsrc(wp, SHARD_BYTES), ((lane >> 4) * (131u * 4096u + 1040u) + (lane & 15)) * 4u, 0, 0);
  const unsigned callv = __hip_atomic_load(callp, __ATOMIC_RELAXED, __HIP_MEMORY_SCOPE_AGENT);
  if (urgent) asm volatile("s_barrier" ::: "memory");
#pragma unroll
  for (int i = 0; i < 7; ++i) pptr[i] = p.peer[prot + i];
  char* lpeer = me;
#pragma unroll
  for (int i = 0; i < 7; ++i) lpeer = lane == (unsigned)i ? p.peer[i] : lpeer;
  rsrc_t pr[7];
  unsigned pidx[7];
#pragma unroll
  for (int i = 0; i < 7; ++i) {
    pidx[i] = prot + i >= 7 ? prot + i - 7 : prot + i;
    pr[i] = mk_rsrc(pptr[i] + CTRL_BYTES, STG_BYTES);
  }
  const rsrc_t own = mk_rsrc(me + CTRL_BYTES, STG_BYTES);
  asm volatile("s_waitcnt lgkmcnt(0)" :: "s"(ka0), "s"(ka1), "s"(ka2), "s"(pr[0]), "s"(pr[1]), "s"(pr[2]), "s"(pr[3]), "s"(pr[4]), "s"(pr[5]), "s"(pr[6]), "v"(lpeer) : "memory");
  __builtin_amdgcn_sched_barrier(0);
  asm volatile("" : "+v"(sx), "+v"(sw));
  sx &= 0x7fffffffu;
  sw &= 0x7fffffffu;
  const unsigned call = rfl(callv) + 1;
  const unsigned par = (call & 1) * PAR_BYTES;
  auto arrive_rdy0 = [&]() {
    unsigned t = 0;
    if (lane == 0) t = __hip_atomic_fetch_add((lu32*)&ctl[2], 1u, __ATOMIC_RELAXED, __HIP_MEMORY_SCOPE_WORKGROUP);
    if (rfl(t) == 3 && lane < 8)
      __hip_atomic_store((gu32*)(lpeer + OFF_RDY0 + ((lane == 7 ? 7u : 6u - lane) * 32u + (blockIdx.x & 31)) * 128u + (is_comm ? 0u : 4u)),
                         call, __ATOMIC_RELAXED, __HIP_MEMORY_SCOPE_SYSTEM);
  };
  {
    constexpr unsigned LO = (127u - 8u) << 23, HI = (127u + 8u) << 23;
    if (__builtin_expect((__ballot(sx >= LO) == 0) | (__ballot(sw >= LO) == 0) | (__ballot(max(sx, sw) >= HI) != 0), 0)) {
      unsigned mx = sx, mw = sw;
#pragma unroll
      for (int o = 1; o < 64; o <<= 1) {
        mx = max(mx, (unsigned)__shfl_xor((int)mx, o));
        mw = max(mw, (unsigned)__shfl_xor((int)mw, o));
      }
      const int ex = (int)(rfl(mx) >> 23), ew = (int)(rfl(mw) >> 23);
      if (ex > 0 && ex < 255 && ew > 0 && ew < 255) {
        const int e = (ew - ex) >> 1;
        const int es = min(max((ctask & 64) ? -e : e, -126), 126);
        const float sc = __builtin_bit_cast(float, (unsigned)(127 + es) << 23);
#pragma unroll
        for (int i = 0; i < 8; ++i) v[i] *= sc;
      }
    }
  }
  {
    const unsigned kb = ctask >> 7;
    if (ctask & 64) {
#pragma unroll
      for (int u = 0; u < 4; ++u) {
        h8 g;
#pragma unroll
        for (int i = 0; i < 8; ++i) g[i] = (_Float16)v[i][u];
        *(__attribute__((address_space(3))) h8*)(cvl + (lane >> 4) * CV_Q + (16 * u + (lane & 15)) * 16) = g;
      }
    } else {
#pragma unroll
      for (int i = 0; i < 8; ++i) {
        h4 g;
#pragma unroll
        for (int u = 0; u < 4; ++u) g[u] = (_Float16)v[i][u];
        *(__attribute__((address_space(3))) h4*)(cvl + ((lane & 7) >> 1) * CV_Q + (8 * i + (lane >> 3)) * 16 + (lane & 1) * 8) = g;
      }
    }
    __builtin_amdgcn_fence(__ATOMIC_SEQ_CST, "wavefront");
    const unsigned cvo = par + (ctask & 64) * (OPER_BYTES / 64) + (ctask & 63) * 1024u + lane * 16u;
    u4 d[4];
#pragma unroll
    for (int q = 0; q < 4; ++q) d[q] = *(__attribute__((address_space(3))) u4*)(cvl + q * CV_Q + lane * 16);
    if (kb == 1) {
#pragma unroll
      for (int q = 0; q < NPUSH; ++q) {
#pragma unroll
        for (int i = 0; i < 7; ++i) {
          __builtin_amdgcn_raw_buffer_store_b128(d[q], pr[i], cvo + (COL_A + 7u * q + 6u - pidx[i]) * COL_BYTES, 0, 17);
          asm volatile("s_waitcnt vmcnt(%0)" :: "i"(PACE) : "memory");
          if (q * 7 + i + 1 == NPUSH * 7 - EARLY) arrive_rdy0();
        }
      }
    }
#pragma unroll
    for (int q = 0; q < 4; ++q) __builtin_amdgcn_raw_buffer_store_b128(d[q], own, cvo + (kb * 4 + q) * COL_BYTES, 0, 17);
    asm volatile("s_waitcnt vmcnt(0)" ::: "memory");
    if (kb == 1) {
      unsigned last = 0;
      if (lane == 0 && __hip_atomic_fetch_add((lu32*)&ctl[3], 1u, __ATOMIC_RELAXED, __HIP_MEMORY_SCOPE_WORKGROUP) == 3)
        last = (__hip_atomic_fetch_add((gu32*)(me + OFF_CNT1), 1u, __ATOMIC_RELAXED, __HIP_MEMORY_SCOPE_AGENT) & 31) == 31;
      if (rfl(last) && lane < 7)
        __hip_atomic_store((gu32*)(lpeer + OFF_PSH + (6u - lane) * 4u), call, __ATOMIC_RELAXED, __HIP_MEMORY_SCOPE_SYSTEM);
    }
    if (kb == 0) arrive_rdy0();
    unsigned last = 0;
    if (lane == 0 && __hip_atomic_fetch_add((lu32*)&ctl[1], 1u, __ATOMIC_RELAXED, __HIP_MEMORY_SCOPE_WORKGROUP) == 7u) {
      __hip_atomic_store(callp, call, __ATOMIC_RELAXED, __HIP_MEMORY_SCOPE_AGENT);
      if ((__hip_atomic_fetch_add((gu32*)(me + OFF_OWN + (blockIdx.x & 7) * 128u), 1u, __ATOMIC_RELAXED, __HIP_MEMORY_SCOPE_AGENT) & 31) == 31)
        last = (__hip_atomic_fetch_add((gu32*)(me + OFF_TOP), 1u, __ATOMIC_RELAXED, __HIP_MEMORY_SCOPE_AGENT) & 7) == 7;
    }
    if (rfl(last) && lane < 8)
      __hip_atomic_store((gu32*)(lpeer + (lane == 7 ? OFF_GO : OFF_RDY + (6u - lane) * 4u)), call, __ATOMIC_RELAXED, __HIP_MEMORY_SCOPE_SYSTEM);
  }
  const rsrc_t ctr = mk_rsrc(me, 0x30000u);

  if (blockIdx.x < NCOMM) {
    if (wid >= 4) return;
    bool ok = true;
    if (wid < 2) {
      const unsigned pw = wid * 32 + blockIdx.x;
      gu32* queue = (gu32*)(me + OFF_QUEUE);
      const unsigned qbase = (call - 1) * (unsigned)NT - 3u * NP;
      auto lookup = [&](unsigned n) -> unsigned {
        const unsigned kg = n / 125u, r = n - kg * 125u, i = n - (NT - 500u);
        unsigned u = (i - 272) >> 2;
        u += (u >= 6) + (u >= 12) + (u >= 18);
        const unsigned t = i < 16 ? (NKG - 4 + (i & 3)) * 128 + (i < 4 ? 60 : (i >> 2) * 7 - 1)
                         : i < 272 ? (NKG - 4 + ((i - 16) >> 6)) * 128 + 64 + ((i - 16) & 63) : (NKG - 4 + (i & 3)) * 128 + u;
        return n >= (unsigned)NT ? 0xffffu : n < NT - 500u ? kg * 128 + (r < 61 ? r : r + 3) : t;
      };
      ok = spin_until(me, call, lane, [&]() {
        unsigned r = 0;
#pragma unroll
        for (int k = 0; k < 4; ++k) {
          const u2 f = __builtin_amdgcn_raw_buffer_load_b64(ctr, OFF_RDY0 + k * 8192u + lane * 128u, 0, 17);
          r |= (f[0] - call) | (f[1] - call);
        }
        return __all((int)r >= 0); });
      const unsigned lbase = par + lane * 16u;
      const unsigned lown = lane < 32 ? lbase + (SHB + 2 * rank) * 256u : 0x40000000u;
      auto unit_off = [&](unsigned task) {
        return (task & 127) == 60 ? lown : lbase + (task & 64) * (OPER_BYTES / 64) + (task & 63) * 1024u;
      };
      auto src_off = [&](unsigned task) {
        return unit_off(task) + ((task >> 7) + (task >= 512 ? NPUSH : 0)) * COL_BYTES + (task == 0xffff ? 0x80000000u : 0u);
      };
      const unsigned roff = OFF_RDY + min(lane, 6u) * 4u;
      const unsigned one = 1;
      unsigned pcol[7];
#pragma unroll
      for (int i = 0; i < 7; ++i) pcol[i] = rfl(pidx[i] * COL_BYTES);
      unsigned prev = 0xffff, cur = ok ? lookup(pw) : 0xffffu;
      bool ready = false;
      u4 v[7];
      {
        const unsigned o = src_off(cur);
#pragma unroll
        for (int i = 0; i < 7; ++i)
          asm volatile("buffer_load_dwordx4 %0, %1, %2, 0 offen sc0 sc1" : "=v"(v[i]) : "v"(o), "s"(pr[i]) : "memory");
      }
      asm volatile("s_waitcnt vmcnt(0)" ::: "memory");
      auto round = [&](unsigned& tk, unsigned& rv) {
        const unsigned od = unit_off(cur) + (COL_B + (cur >> 7) * 7u) * COL_BYTES;
        asm volatile("s_waitcnt vmcnt(15)" : "+v"(tk), "+v"(rv) :: "memory");
        unsigned nxt = lookup(rfl(tk) - qbase);
        if (!ready) {
          ready = __all((int)(rv - call) >= 0);
          if (!ready && nxt != 0xffff && nxt >= 4 * 128) {
            ok = spin_until(me, call, lane, [&]() {
              return __all((int)(__builtin_amdgcn_raw_buffer_load_b32(ctr, roff, 0, 17) - call) >= 0); });
            ready = true;
            if (!ok) nxt = 0xffff;
          }
        }
        const unsigned os = src_off(nxt);
#pragma unroll
        for (int i = 0; i < 7; ++i) {
          if (i) asm volatile("s_waitcnt vmcnt(15)" ::: "memory");
          asm volatile("buffer_store_dwordx4 %0, %1, %2, %3 offen sc1\n\tbuffer_load_dwordx4 %0, %4, %5, 0 offen sc0 sc1"
                       : "+v"(v[i]) : "v"(od), "s"(own), "s"(pcol[i]), "v"(os), "s"(pr[i]) : "memory");
        }
        const unsigned fo = prev == 0xffff ? OFF_DUMMY + pw * 64u : OFF_FB + (prev & 127) * FB_U + (prev >> 7) * 4u;
        if (lane == 0) {
          asm volatile("buffer_store_dword %0, %1, %2, 0 offen sc1" :: "v"(call), "v"(fo), "s"(ctr) : "memory");
          asm volatile("global_atomic_add %0, %1, %2, off sc0" : "+v"(tk) : "v"(queue), "v"(one) : "memory");
        }
        asm volatile("buffer_load_dword %0, %1, %2, 0 offen sc0 sc1" : "+v"(rv) : "v"(roff), "s"(ctr) : "memory");
        prev = cur; cur = nxt;
      };
      unsigned tka = qbase + NP + pw, tkb = qbase + 2 * NP + pw, rva = call - 1, rvb = call - 1;
      while (cur != 0xffff) {
        round(tka, rva);
        if (cur == 0xffff) break;
        round(tkb, rvb);
      }
      asm volatile("s_waitcnt vmcnt(4)" ::: "memory");
      if (lane == 0 && prev != 0xffff)
        __hip_atomic_store((gu32*)(me + OFF_FB + (prev & 127) * FB_U + (prev >> 7) * 4u), call, __ATOMIC_RELAXED, __HIP_MEMORY_SCOPE_AGENT);
      asm volatile("s_waitcnt vmcnt(0)" :: "v"(tka), "v"(tkb), "v"(rva), "v"(rvb), "v"(v[0]), "v"(v[1]), "v"(v[2]), "v"(v[3]),
                   "v"(v[4]), "v"(v[5]), "v"(v[6]) : "memory");
      if (lane == 0) ctl[3] = 1;
    } else {
      unsigned code_len;
      asm volatile("s_mov_b32 %0, .Lgemm_ar_code_end-.Lgemm_ar_code_begin" : "=s"(code_len));
      code_len = (((unsigned)pc_entry + code_len) | 4095u) + 1u - (unsigned)pc_entry;
      const rsrc_t code = mk_rsrc((const void*)pc_entry, code_len);
      for (unsigned n = 0; rfl(ctl[3]) == 0; ++n) {
        if ((n & 15) == 0 && blockIdx.x < 8 && wid == 2) {
          unsigned t = 0;
#pragma unroll
          for (int k = 0; k < 5; ++k) t |= __builtin_amdgcn_raw_buffer_load_b32(code, lane * 128u, k * 8192u, 16);
          asm volatile("" :: "v"(t) : "memory");
        }
        __builtin_amdgcn_s_sleep(16);
      }
    }
    if (blockIdx.x < 28) {
      const unsigned item = wid * 28 + blockIdx.x, otc = item / 7, pi = item - otc * 7;
      const unsigned so = (SHB + 2 * ((rank + 1 + pi) & (NR - 1))) * (16u * 16384u);
      const rsrc_t po = mk_rsrc(p.pout[pi] + so, 32u * 16384u);
      const rsrc_t ro = mk_rsrc((char*)outp + so, 32u * 16384u);
      if (ok)
        ok = spin_until(me, call, lane, [&]() {
          return (int)(rfl(__builtin_amdgcn_raw_buffer_load_b32(ctr, OFF_OUTF + (pi * 16u + otc) * 128u, 0, 17)) - call) >= 0; });
      const unsigned o = otc * 1024u + lane * 16u;
      if (ok) {
        u4 w[OD];
#pragma unroll
        for (int i = 0; i < OD; ++i)
          asm volatile("buffer_load_dwordx4 %0, %1, %2, %3 offen sc0 sc1" : "=v"(w[i]) : "v"(o), "s"(po), "s"(i * 16384u) : "memory");
#pragma unroll
        for (int i = 0; i < OD; ++i)
          asm volatile("s_waitcnt vmcnt(%6)\n\tbuffer_store_dwordx4 %0, %1, %2, %3 offen sc1\n\tbuffer_load_dwordx4 %0, %1, %4, %5 offen sc0 sc1"
                       : "+v"(w[i]) : "v"(o), "s"(ro), "s"(i * 16384u), "s"(po), "s"((i + OD) * 16384u), "i"(OD - 1 + i) : "memory");
        for (unsigned r = OD; r < 32; r += OD) {
#pragma unroll
          for (int i = 0; i < OD; ++i)
            asm volatile("s_waitcnt vmcnt(%6)\n\tbuffer_store_dwordx4 %0, %1, %2, %3 offen sc1\n\tbuffer_load_dwordx4 %0, %1, %4, %5 offen sc0 sc1"
                         : "+v"(w[i]) : "v"(o), "s"(ro), "s"((r + i) * 16384u), "s"(po), "s"((r + i + OD) * 16384u), "i"(2 * OD - 2) : "memory");
        }
        asm volatile("s_waitcnt vmcnt(0)" :: "v"(w[0]), "v"(w[1]), "v"(w[2]), "v"(w[3]), "v"(w[OD - 4]), "v"(w[OD - 3]), "v"(w[OD - 2]), "v"(w[OD - 1]) : "memory");
      } else {
        const u4 nan4 = {0x7fc00000u, 0x7fc00000u, 0x7fc00000u, 0x7fc00000u};
        for (int r = 0; r < 32; ++r) __builtin_amdgcn_raw_buffer_store_b128(nan4, ro, o + r * 16384u, 0, 16);
      }
    }
    return;
  }

  const unsigned xcd = gi & 7, ti7 = gi >> 5, ti = (xcd & 1) * 7 + ti7;
  const unsigned tp = xcd >> 1, tq = (gi >> 3) & 3;
  const unsigned tc = tq * 4 + ((tp + tq) & 3), col0 = tc * TN;
  const bool tall = ti7 >= 5;
  const unsigned p8 = tall ? 224 + 2 * ((xcd & 1) * 2 + ti7 - 5) : 232 + (xcd & 1) * 5 + ti7;
  auto blk = [&](unsigned g, unsigned m) -> unsigned {
    const unsigned l = m < 8 ? 28 * m + 2 * ti + g : p8 + (tall ? g : 0u);
    return l < 2 ? SHB + 2 * rank + l : l - 2;
  };

  const rsrc_t stg = mk_rsrc(me + CTRL_BYTES + par, STG_BYTES - PAR_BYTES);
  constexpr int ND0 = 9;
  const unsigned pc0 = min(wid * ND0, 34u - ND0);
  unsigned dvo[ND0];
#pragma unroll
  for (int i = 0; i < ND0; ++i) {
    const unsigned pc = pc0 + i;
    if (pc < 18) {
      const unsigned e = pc * 64 + lane, q = e / TM, r = e - q * TM, g = r >= 144, m = (r - 144 * g) >> 4;
      dvo[i] = q * COL_BYTES + (blk(g, m) * 16u + (r & 15)) * 16u;
    } else {
      const unsigned e = (pc - 18) * 64 + lane, q = e >> 8, r = e & 255;
      dvo[i] = OPER_BYTES + q * COL_BYTES + (col0 + r) * 16u;
    }
  }
  const unsigned ldsb = (unsigned)(size_t)lds;
  auto dma = [&](unsigned m0, unsigned vo, unsigned so) __attribute__((always_inline)) {
    asm volatile("s_mov_b32 m0, %0\n\ts_nop 0\n\tbuffer_load_dwordx4 %1, %2, %3 offen sc1 lds"
                 :: "s"(m0), "v"(vo), "s"(stg), "s"(so) : "memory");
  };

  unsigned poff;
  {
    const unsigned wk = min(lane / 14, 3u), wh = lane % 14;
    const unsigned u = wh < 10 ? min(blk(0, wh) >> 2, 60u) : 64 + tc * 4 + (wh - 10);
    poff = OFF_FB + u * FB_U + wk * 4u;
  }
  unsigned kg0 = 0;
  unsigned avail = COL_A;
  bool ok = true;
  auto poll_issue = [&]() -> unsigned { return __builtin_amdgcn_raw_buffer_load_b32(ctr, poff, kg0 * 4u, 16); };
  auto poll_eval = [&](unsigned v) {
    const unsigned long long m = __ballot((int)(v - call) >= 0) & 0x00ffffffffffffffull;
    kg0 = min(kg0 + (unsigned)__builtin_ctzll(~m) / 14u, (unsigned)NKG - 4);
    avail = COL_B + kg0 * 7;
  };

  const unsigned wr = wid >> 2, wc = wid & 3;
  const unsigned a_off = (lane >> 4) * A_COL + (wr * 144 + (lane & 15)) * 16;
  const unsigned b_off = A_BYTES + (lane >> 4) * B_COL + (wc * 64 + (lane & 15)) * 16;
  f4 acc[9][4];
#pragma unroll
  for (int m = 0; m < 9; ++m)
#pragma unroll
    for (int n = 0; n < 4; ++n) acc[m][n] = f4{0.f, 0.f, 0.f, 0.f};

  unsigned pv = 0, pend = 0;
  if (wid == 7)
    ok = spin_until(me, call, lane, [&]() { return (int)(rfl(__builtin_amdgcn_raw_buffer_load_b32(ctr, OFF_GO, 0, 17)) - call) >= 0; });
  asm volatile("s_barrier" ::: "memory");
  unsigned jev = COL_A / 4 - 4;
  auto poll_step = [&](unsigned j) __attribute__((always_inline)) {
    const unsigned need = (j + (j < JSW ? 4 : 3)) * 4;
    if (!ok) {
    } else if (avail == COL_A) {
      if (need > COL_A) {
        ok = spin_until(me, call, lane, [&]() {
          return __all((int)(__builtin_amdgcn_raw_buffer_load_b32(ctr, OFF_PSH + min(lane, 6u) * 4u, 0, 17) - call) >= 0); });
        avail = COL_B;
      }
    } else {
      if (pend) poll_eval(pv);
      if (avail < need) ok = spin_until(me, call, lane, [&]() { poll_eval(poll_issue()); return avail >= need; });
    }
    asm volatile("s_waitcnt lgkmcnt(0)" ::: "memory");
    const unsigned jblk = avail / 4 - (avail / 4 < JSW + 3 ? 3 : 2);
    pend = 0;
    if (!ok || avail >= COL_T) {
      jev = 1000;
    } else if (avail != COL_A && jblk <= j + 4) {
      pv = poll_issue(); pend = 1;
      jev = min(jblk, j + 2);
    } else {
      jev = avail == COL_A ? jblk : jblk - 4;
    }
  };
  const unsigned la = ldsb + a_off, lb = ldsb + b_off;
  h8 a[9], b[4];
  auto kloop = [&](auto mu_c, auto role_c) __attribute__((always_inline)) {
    constexpr int MU = decltype(mu_c)::value;
    constexpr int ROLE = decltype(role_c)::value;
#define RD(dst, base, off) asm volatile("ds_read_b128 %0, %1 offset:%2" : "+v"(dst) : "v"(base), "i"(off) : "memory")
#define SB __builtin_amdgcn_sched_barrier(0)
    const unsigned lds0 = ldsb + pc0 * 1024u;
    if (ROLE == 0) {
#pragma unroll
      for (int st = 0; st < 3; ++st)
#pragma unroll
        for (int i = 0; i < ND0; ++i) dma(lds0 + st * STAGE + i * 1024u, dvo[i], st * (4u * COL_BYTES));
      asm volatile("s_waitcnt vmcnt(%0)" :: "i"(2 * ND0) : "memory");
    }
    asm volatile("s_barrier" ::: "memory");
    RD(b[0], lb, 0); RD(b[1], lb, 256);
#pragma unroll
    for (int m = 0; m < MU; ++m) RD(a[m], la, m * 256);
    RD(b[2], lb, 512); RD(b[3], lb, 768);
    if (ROLE != 0) __builtin_amdgcn_s_setprio(1);
    for (unsigned j = 0; j < NMAIN; ++j) {
      if (ROLE == 2 && __builtin_expect(j >= jev, 0)) poll_step(j);
      if (ROLE != 0) asm volatile("s_barrier" ::: "memory");
      const unsigned sd = j < JSW ? j + 3 : j + 2;
      const unsigned m0 = lds0 + (sd & 3) * STAGE;
      const unsigned so = sd * (4u * COL_BYTES);
      const unsigned nb = ((j + 1) & 3) * STAGE;
      const unsigned na = la + nb, nbb = lb + nb;
      SB;
#pragma unroll
      for (int m = 0; m < MU; ++m) {
        if (m == 0) asm volatile("s_waitcnt lgkmcnt(%3)" : "+v"(a[0]), "+v"(b[0]), "+v"(b[1]) : "i"(MU + 1) : "memory");
        else asm volatile("s_waitcnt lgkmcnt(%1)" : "+v"(a[m]) : "i"(MU + 1 - m) : "memory");
        acc[m][0] = __builtin_amdgcn_mfma_f32_16x16x32_f16(a[m], b[0], acc[m][0], 0, 0, 0);
        acc[m][1] = __builtin_amdgcn_mfma_f32_16x16x32_f16(a[m], b[1], acc[m][1], 0, 0, 0);
        SB;
      }
      if (ROLE == 0) {
        if (j > JSW) asm volatile("s_waitcnt vmcnt(0)" ::: "memory");
        asm volatile("s_waitcnt vmcnt(%0)\n\ts_barrier" :: "i"(ND0) : "memory");
      }
      RD(b[0], nbb, 0); RD(b[1], nbb, 256);
      if (ROLE == 0 && j != JSW && sd < NMAIN) {
#pragma unroll
        for (int i = 0; i < ND0; ++i) dma(m0 + i * 1024u, dvo[i], so);
      }
      asm volatile("s_waitcnt lgkmcnt(2)" : "+v"(b[2]), "+v"(b[3]) :: "memory");
      SB;
#pragma unroll
      for (int m = 0; m < MU; ++m) {
        acc[m][2] = __builtin_amdgcn_mfma_f32_16x16x32_f16(a[m], b[2], acc[m][2], 0, 0, 0);
        acc[m][3] = __builtin_amdgcn_mfma_f32_16x16x32_f16(a[m], b[3], acc[m][3], 0, 0, 0);
        SB;
        RD(a[m], na, m * 256);
        SB;
      }
      RD(b[2], nbb, 512); RD(b[3], nbb, 768);
      SB;
    }
    asm volatile("s_waitcnt lgkmcnt(0)" :: "v"(a[0]), "v"(a[1]), "v"(a[2]), "v"(a[3]), "v"(a[4]), "v"(a[5]),
                 "v"(a[6]), "v"(a[7]), "v"(a[MU - 2]), "v"(a[MU - 1]), "v"(b[0]), "v"(b[1]), "v"(b[2]), "v"(b[3]) : "memory");
    if (ROLE != 0) __builtin_amdgcn_s_setprio(0);
  };
  if (wid < 4) kloop(IC<9>{}, IC<0>{});
  else if (!tall && wid < 7) kloop(IC<8>{}, IC<1>{});
  else if (!tall) kloop(IC<8>{}, IC<2>{});
  else if (wid < 7) kloop(IC<9>{}, IC<1>{});
  else kloop(IC<9>{}, IC<2>{});

  {
    unsigned have = 0;
    auto tail_eval = [&](unsigned v) {
      const unsigned long long m = __ballot((int)(v - call) >= 0);
      const unsigned h3 = (unsigned)(m & (m >> 14) & (m >> 28));
      have = (h3 & (unsigned)(m >> 42) & 0x3fffu) | ((h3 & 0x3c00u) << 4);
    };
    auto tail_wait = [&](unsigned mask) __attribute__((always_inline)) {
      if (__builtin_expect(ok && (have & mask) != mask, 0))
        ok = spin_until(me, call, lane, [&]() {
          tail_eval(__builtin_amdgcn_raw_buffer_load_b32(ctr, poff, (NKG - 4) * 4u, 16)); return (have & mask) == mask; });
    };
    const bool dmaw = wid < 7;
    const unsigned wvo = OPER_BYTES + (col0 + lane) * 16u;
    auto dma_w = [&](unsigned c) __attribute__((always_inline)) {
#pragma unroll
      for (int i = 0; i < 4; ++i) dma(ldsb + c * B_COL + i * 1024u, wvo, (COL_T + c) * COL_BYTES + i * 1024u);
    };
    if (wid == 7) tail_wait(0x3c000u);
    asm volatile("s_barrier" ::: "memory");
    if (dmaw) { dma_w(wid); dma_w(7 + wid); dma_w(14 + wid); }
    else tail_wait(0x3c01u);
    asm volatile("s_barrier" ::: "memory");
    const unsigned xg = (lane >> 4) & 1;
    const unsigned xvo = (COL_T + wid * 4 + (lane >> 5)) * COL_BYTES + (lane & 15) * 16u;
    auto xreg = [](int m) -> unsigned {
      return m < 3 ? TW_BYTES + m * TX_BYTES : m == 3 ? TW_BYTES : m == 4 ? 6 * 16384u : m == 5 ? 5 * 16384u
           : m == 6 ? TW_BYTES + TX_BYTES : m == 7 ? 4 * 16384u : m == 8 ? TW_BYTES + 2 * TX_BYTES : 3 * 16384u;
    };
    auto dma_x = [&](int m) __attribute__((always_inline)) {
      const unsigned vo = xvo + blk(xg, m) * 256u;
      const unsigned m0 = ldsb + xreg(m) + wid * 2048u;
      dma(m0, vo, 0u);
      dma(m0 + 1024u, vo, 2u * COL_BYTES);
    };
    if (dmaw) { dma_x(0); dma_w(21 + wid); }
    else tail_wait(2u);
    if (wid == 7 && lane == 0) ctl[0] = ok ? 0u : 1u;
    asm volatile("s_waitcnt vmcnt(0) lgkmcnt(0)\n\ts_barrier" ::: "memory");
    const rsrc_t ro = mk_rsrc(outp, OUT_BYTES);
    const unsigned ovo = (4 * (lane >> 4) * 4096u + col0 + wc * 64 + 4 * (lane & 15)) * 4u;
    const int mcnt = wr == 0 || tall ? 9 : 8;
    typedef __attribute__((address_space(3))) h8 lh8;
    lchar* fb = lds + (lane >> 4) * B_COL + (wc * 64 + (lane & 15)) * 16;
    lchar* fb4 = fb + 16 * B_COL;
    asm volatile("" : "+v"(fb4));
    lchar* fa = lds + TW_BYTES + (lane >> 4) * 512 + wr * 256 + (lane & 15) * 16;
    lchar* fa3 = fa - TW_BYTES + 3 * 16384;
    asm volatile("" : "+v"(fa3));
    h8 bw[7][4];
#pragma unroll
    for (int m = 0; m < 9; ++m) {
      if (dmaw) {
        if (m == 0) dma_x(1);
        else if (m < 4) { dma_x(2 * m); dma_x(2 * m + 1); }
        else if (m == 4) dma_x(8);
      }
      if (m < mcnt) {
        const unsigned badv = ctl[0];
        h8 af[7];
        const int R = m == 0 ? 0 : m + 2 < 7 ? m + 2 : 7;
        auto ord = [&](int i) { return i < R ? 6 - i : i - R; };
        auto fetch = [&](int s) __attribute__((always_inline)) {
          af[s] = xreg(m) >= TW_BYTES ? *(lh8*)(fa + (xreg(m) - TW_BYTES) + s * 2048) : *(lh8*)(fa3 + (xreg(m) - 3 * 16384) + s * 2048);
          if (s < 7 - R) {
#pragma unroll
            for (int n = 0; n < 4; ++n) bw[s][n] = *(lh8*)((s < 4 ? fb : fb4) + (s & 3) * 4 * B_COL + n * 256);
          }
        };
        constexpr int WIN = TWIN;
#pragma unroll
        for (int i = 0; i < WIN; ++i) fetch(ord(i));
#pragma unroll
        for (int i = 0; i < 7; ++i) {
          if (i + WIN < 7) fetch(ord(i + WIN));
          SB;
#pragma unroll
          for (int n = 0; n < 4; ++n)
            acc[m][n] = __builtin_amdgcn_mfma_f32_16x16x32_f16(af[ord(i)], bw[ord(i)][n], acc[m][n], 0, 0, 0);
          SB;
        }
        const bool bad = rfl(badv) != 0;
        if (m < 4 && wid == 7) {
          tail_wait(12u << (2 * m));
          if (__builtin_expect(!ok, 0) && lane == 0) ctl[0] = 1u;
        }
        SB;
        const unsigned orow = ovo + blk(wr, m) * (16u * 16384u);
#pragma unroll
        for (int j = 0; j < 4; ++j) {
          f4 o = f4{acc[m][0][j], acc[m][1][j], acc[m][2][j], acc[m][3][j]};
          if (bad) o = f4{__builtin_nanf(""), __builtin_nanf(""), __builtin_nanf(""), __builtin_nanf("")};
          __builtin_amdgcn_raw_buffer_store_b128(__builtin_bit_cast(u4, o), ro, orow + j * 16384u, 0, 17);
        }
      }
      if (m == 0 && ti == 0) {
        asm volatile("s_waitcnt vmcnt(0)" ::: "memory");
        unsigned t = 0;
        if (lane == 0) t = __hip_atomic_fetch_add((lu32*)&ctl[1], 1u, __ATOMIC_RELAXED, __HIP_MEMORY_SCOPE_WORKGROUP);
        if (rfl(t) == 15 && lane < 7) {
          char* fp = p.peer[0];
#pragma unroll
          for (int i = 1; i < 7; ++i) fp = lane == (unsigned)i ? p.peer[i] : fp;
          __hip_atomic_store((gu32*)(fp + OFF_OUTF + ((6u - lane) * 16u + tc) * 128u), call, __ATOMIC_RELAXED, __HIP_MEMORY_SCOPE_SYSTEM);
        }
      }
      if (m < 4 || m == 5)
        asm volatile("s_waitcnt vmcnt(%0) lgkmcnt(0)\n\ts_barrier" :: "i"(m == 0 ? 4 : m == 1 ? 6 : m < 4 ? 12 : 8) : "memory");
      SB;
    }
  }
  asm volatile(".Lgemm_ar_code_end:");
}

}

extern "C" void kernel_launch(int rank, int nranks, ncclComm_t comm,
                              hipStream_t stream,
                              void* const* d_in, const int* in_sizes, int n_in,
                              void* d_out, int out_size,
                              const int* in_dtypes, int out_dtype,
                              void* const* d_out_all,
                              void* const* d_ws_all, size_t ws_bytes) {
  if (nranks != NR || (unsigned)rank >= (unsigned)NR || ws_bytes < WS_NEED || n_in != 2 ||
      in_sizes[0] != 4096 * 512 || in_sizes[1] != 4096 * 512 || out_size != 4096 * 4096) {
    fprintf(stderr, "kernel_launch: unexpected problem shape\n");
    return;
  }
  Params p;
  p.x = (const float*)d_in[0];
  p.w = (const float*)d_in[1];
  p.out = (float*)d_out;
  p.me = (char*)d_ws_all[rank];
  for (int i = 0; i < 7; ++i) {
    p.peer[i] = p.peer[7 + i] = (char*)d_ws_all[(rank + 1 + i) % NR];
    p.pout[i] = (char*)d_out_all[(rank + 1 + i) % NR];
  }
  p.rank = (unsigned)rank;
  const size_t sb = CTRL_BYTES + (SHB + 2 * ((rank + 1) % NR)) * 256u;
  for (int i = 0; i < 8; ++i) {
    const char* ws = i < 7 ? p.peer[i] : p.me;
    p.warm[i] = ws + sb + (size_t)COL_A * COL_BYTES;
    p.warm[8 + i] = ws + OFF_DUMMY;
    p.warm[16 + i] = ws + sb + (size_t)32 * COL_BYTES;
    p.warm[24 + i] = (i < 7 ? p.pout[i] : (char*)p.out) + (size_t)SHB * 16 * 16384;
  }
  for (int i = 0; i < 16; ++i) {
    p.warm[32 + i] = (char*)p.out + (size_t)i * (4u << 20);
    p.warm[48 + i] = p.me + sb + (size_t)i * 32 * COL_BYTES;
  }
  hipLaunchKernelGGL(gemm_ar_kernel, dim3(NWG), dim3(512), 0, stream, p);
  HIP_CHECK(hipGetLastError());
}
```
